```python
import numpy as np

kernel_name = "gpu_ag_gemm_m4096_k4096_n4096_f32_reluq8"


ABSMAX_RTOL = 0.02

COMPUTE_DTYPE = "float32"
EPILOGUE = "relu_quant_int8"
IN_QUANT = None
OUT_QUANT = "int8"
IN_DTYPES = {"x": "float32", "w_mat": "float32"}
WORLD_SIZE = 8

def setup_inputs():

    x = np.random.randn(4096, 4096).astype(np.float32)
    w_mat = (np.random.randn(4096, 4096) / np.sqrt(4096)).astype(np.float32)
    return {"x": x, "w_mat": w_mat}

def _quant_dequant(y, amax):
    scale = amax / 127.0
    q = np.clip(np.round(y / scale), -127.0, 127.0)
    return (q * scale).astype(np.float32)
def _epilogue(y):
    y = np.maximum(y, 0.0)
    return _quant_dequant(y, np.abs(y).max())

def reference(x, w_mat):
    y = (x @ w_mat).astype(np.float32)
    return _epilogue(y)
```

```cpp
#include <hip/hip_runtime.h>
#include <rccl/rccl.h>
#include <cstdio>

#define HIP_CHECK(expr) do { hipError_t _e = (expr); if (_e != hipSuccess) { \
    fprintf(stderr, "HIP error %s at %s:%d\n", hipGetErrorString(_e), __FILE__, __LINE__); \
    return; } } while (0)

typedef unsigned v4u __attribute__((ext_vector_type(4)));
typedef float f32x4 __attribute__((ext_vector_type(4)));
typedef float f32x2 __attribute__((ext_vector_type(2)));
typedef __bf16 bf16x2 __attribute__((ext_vector_type(2)));
typedef short bf16x8 __attribute__((ext_vector_type(8)));
typedef unsigned long long u64;

constexpr int NR = 8;
constexpr int KD = 4096;
constexpr int SH = 512;
constexpr int NWG = 256;
constexpr int NTHR = 320;
constexpr unsigned XS_OFF = 0;
constexpr unsigned WS_OFF = 16u << 20;
constexpr unsigned FLAG_OFF = 24u << 20;
constexpr unsigned SLOT1_OFF = FLAG_OFF + (512u << 10);
constexpr unsigned SLOT2_OFF = SLOT1_OFF + (16u << 10);
constexpr unsigned DONE_OFF = SLOT2_OFF + (4u << 10);
constexpr unsigned CNT_OFF = DONE_OFF + (4u << 10);
constexpr unsigned TMO_OFF = CNT_OFF + (16u << 10);
constexpr unsigned WS_USED = TMO_OFF + 4096;
constexpr unsigned STAMP_OFF = 32u << 20;
constexpr u64 DEADLINE_TICKS = 50000000ull;
constexpr int LDS_BYTES = 96 * 1024;
constexpr int AUX_SYS = 17;

struct Params {
  const float* x;
  const float* w;
  char* ws[NR];
  char* out[NR];
  int rank;
};

#define RSRC(p, bytes) __builtin_amdgcn_make_buffer_rsrc((void*)(p), 0, (int)(bytes), 0x00020000)
#define CBAR() asm volatile("" ::: "memory")
#define WAITV(n) asm volatile("s_waitcnt vmcnt(" #n ")" ::: "memory")

#define STAMP(i) do {} while (0)

__device__ __forceinline__ unsigned pk(float a, float b) {
  f32x2 v = {a, b};
  return __builtin_bit_cast(unsigned, __builtin_convertvector(v, bf16x2));
}

__device__ __forceinline__ void unit_load(v4u (&r)[8], __amdgpu_buffer_rsrc_t rx, __amdgpu_buffer_rsrc_t rw,
                                          bool isw, int blk, int k32, int lane) {
  const int kg = lane >> 4, c = lane & 15;
  if (!isw) {
    const unsigned base = (unsigned)(blk * 64 + c) * (KD * 4) + (unsigned)(k32 * 32 + 4 * kg) * 4;
#pragma unroll
    for (int m = 0; m < 4; ++m) {
      r[2 * m] = __builtin_amdgcn_raw_buffer_load_b128(rx, base + m * (16 * KD * 4), 0, 0);
      r[2 * m + 1] = __builtin_amdgcn_raw_buffer_load_b128(rx, base + m * (16 * KD * 4) + 64, 0, 0);
    }
  } else {
    const unsigned base = (unsigned)(k32 * 32 + 4 * kg) * (SH * 4) + (unsigned)(blk * 64 + 4 * c) * 4;
#pragma unroll
    for (int jj = 0; jj < 8; ++jj)
      r[jj] = __builtin_amdgcn_raw_buffer_load_b128(rw, base + ((jj & 3) + 16 * (jj >> 2)) * (SH * 4), 0, 0);
  }
}
__device__ __forceinline__ float asf(unsigned u) { return __builtin_bit_cast(float, u); }
__device__ __forceinline__ void unit_cvt(v4u (&o)[4], const v4u (&r)[8], bool isw) {
  if (!isw) {
#pragma unroll
    for (int m = 0; m < 4; ++m) {
      o[m].x = pk(asf(r[2 * m].x), asf(r[2 * m].y));
      o[m].y = pk(asf(r[2 * m].z), asf(r[2 * m].w));
      o[m].z = pk(asf(r[2 * m + 1].x), asf(r[2 * m + 1].y));
      o[m].w = pk(asf(r[2 * m + 1].z), asf(r[2 * m + 1].w));
    }
  } else {
#pragma unroll
    for (int i = 0; i < 4; ++i) {
      o[i].x = pk(asf(r[0][i]), asf(r[1][i]));
      o[i].y = pk(asf(r[2][i]), asf(r[3][i]));
      o[i].z = pk(asf(r[4][i]), asf(r[5][i]));
      o[i].w = pk(asf(r[6][i]), asf(r[7][i]));
    }
  }
}
__device__ __forceinline__ unsigned stage_off(int dblk, int k32) {
  const int kc = k32 >> 1, h = k32 & 1;
  return dblk < 32 ? XS_OFF + (unsigned)((kc * 32 + dblk) * 2 + h) * 4096u
                   : WS_OFF + (unsigned)((kc * 16 + (dblk - 32)) * 2 + h) * 4096u;
}
__device__ __forceinline__ unsigned flag_off(int dblk, int k32) {
  return FLAG_OFF + (unsigned)(dblk * 64 + (k32 >> 1)) * 128u + (unsigned)(k32 & 1) * 4u;
}
__device__ __forceinline__ void unit_store(const v4u (&o)[4], __amdgpu_buffer_rsrc_t rd, unsigned off, int lane) {
#pragma unroll
  for (int i = 0; i < 4; ++i) __builtin_amdgcn_raw_buffer_store_b128(o[i], rd, off + lane * 16 + i * 1024, 0, AUX_SYS);
}

__device__ __forceinline__ bool spin_fail(u64& t0, unsigned* tmo, unsigned call) {
  const u64 now = __builtin_amdgcn_s_memrealtime();
  if (t0 == 0) t0 = now;
  if (now - t0 > 2000) {
    if (__hip_atomic_load(tmo, __ATOMIC_RELAXED, __HIP_MEMORY_SCOPE_AGENT) == call) return true;
    if (now - t0 > DEADLINE_TICKS) {
      __hip_atomic_store(tmo, call, __ATOMIC_RELAXED, __HIP_MEMORY_SCOPE_AGENT);
      return true;
    }
  }
  return false;
}

__global__ void __launch_bounds__(NTHR, 2) ag_gemm_kernel(Params p) {
  extern __shared__ __attribute__((aligned(16))) char smem[];
  const int lane = threadIdx.x & 63;
  const int wid = __builtin_amdgcn_readfirstlane(threadIdx.x >> 6);
  const int b = blockIdx.x;
  const int rank = p.rank, g = rank >> 2, jq = rank & 3;
  const int xcd = b & 7, bi = b >> 3;
  char* const myws = p.ws[rank];
  unsigned* const tmo = (unsigned*)(myws + TMO_OFF);
  unsigned* const cntp = (unsigned*)(myws + CNT_OFF + b * 64);
  const __amdgpu_buffer_rsrc_t rx = RSRC(p.x, SH * KD * 4);
  const __amdgpu_buffer_rsrc_t rw = RSRC(p.w, SH * KD * 4);
  STAMP(0);

  if (wid == 4) {
    const int dsel = bi & 7;
    if (dsel < 7) {
      const int dst = (rank + 1 + dsel) & 7;
      const bool same = (dst >> 2) == g;
      const int v = xcd + 8 * (bi >> 3);
      const __amdgpu_buffer_rsrc_t rd = RSRC(p.ws[dst], WS_USED);
      auto decode = [&](int u, bool& isw, int& sblk, int& dblk, int& k32) {
        k32 = u / 5;
        const int s = u - k32 * 5;
        if (same) { isw = false; sblk = s; dblk = jq * 5 + s; }
        else if (s < 3) { isw = false; sblk = 5 + s; dblk = 20 + jq * 3 + s; }
        else { isw = true; sblk = (dst & 3) * 2 + (s - 3); dblk = 32 + 8 + jq * 2 + (s - 3); }
      };
      v4u ra[8], rb2[8], o[4];
      bool isw; int sblk, dblk, k32;
      decode(v, isw, sblk, dblk, k32);
      unit_load(ra, rx, rw, isw, sblk, k32, lane);
      const unsigned call = __hip_atomic_load(cntp, __ATOMIC_RELAXED, __HIP_MEMORY_SCOPE_AGENT) + 1;
      unsigned pflag = 0xffffffffu;
#pragma unroll 1
      for (int t = 0; t < 20; t += 2) {
        bool isw2; int sblk2, dblk2, k322;
        decode(v + 32 * (t + 1), isw2, sblk2, dblk2, k322);
        unit_load(rb2, rx, rw, isw2, sblk2, k322, lane);
        WAITV(8);
        if (lane == 0) __builtin_amdgcn_raw_buffer_store_b32(call, rd, pflag, 0, AUX_SYS);
        unit_cvt(o, ra, isw);
        unit_store(o, rd, stage_off(dblk, k32), lane);
        pflag = flag_off(dblk, k32);
        if (t + 2 < 20) {
          decode(v + 32 * (t + 2), isw, sblk, dblk, k32);
          unit_load(ra, rx, rw, isw, sblk, k32, lane);
          WAITV(8);
        } else {
          WAITV(0);
        }
        if (lane == 0) __builtin_amdgcn_raw_buffer_store_b32(call, rd, pflag, 0, AUX_SYS);
        unit_cvt(o, rb2, isw2);
        unit_store(o, rd, stage_off(dblk2, k322), lane);
        pflag = flag_off(dblk2, k322);
      }
      WAITV(0);
      if (lane == 0) __builtin_amdgcn_raw_buffer_store_b32(call, rd, pflag, 0, AUX_SYS);
      STAMP(1);
      return;
    }
    if (bi != 7) return;
    const unsigned call = __hip_atomic_load(cntp, __ATOMIC_RELAXED, __HIP_MEMORY_SCOPE_AGENT) + 1;
    WAITV(0);
    __builtin_amdgcn_s_barrier();
    STAMP(1);
    {
      const u64* s1 = (const u64*)(myws + SLOT1_OFF + xcd * 1024) + lane * 2;
      u64 t0 = 0;
      unsigned m = 0;
      for (;;) {
        const u64 a = __hip_atomic_load(s1, __ATOMIC_RELAXED, __HIP_MEMORY_SCOPE_AGENT);
        const u64 c = __hip_atomic_load(s1 + 1, __ATOMIC_RELAXED, __HIP_MEMORY_SCOPE_AGENT);
        m = max((unsigned)a, (unsigned)c);
        if (__all((unsigned)(a >> 32) == call && (unsigned)(c >> 32) == call)) break;
        if (spin_fail(t0, tmo, call)) break;
      }
#pragma unroll
      for (int s = 32; s > 0; s >>= 1) m = max(m, (unsigned)__shfl_xor((int)m, s));
      STAMP(2);
      if (lane < NR) {
        char* dws = p.ws[0];
#pragma unroll
        for (int q = 1; q < NR; ++q) dws = (lane == q) ? p.ws[q] : dws;
        __hip_atomic_store((u64*)(dws + SLOT2_OFF) + rank * 8 + xcd, ((u64)call << 32) | m, __ATOMIC_RELAXED,
                           __HIP_MEMORY_SCOPE_SYSTEM);
      }
    }
    {
      const unsigned* dn = (const unsigned*)(myws + DONE_OFF) + xcd * 64 + lane;
      u64 t0 = 0;
      bool bad = false;
      for (;;) {
        const unsigned a = __hip_atomic_load(dn, __ATOMIC_RELAXED, __HIP_MEMORY_SCOPE_SYSTEM);
        if (__all(a >= call)) break;
        if (spin_fail(t0, tmo, call)) { bad = true; break; }
        __builtin_amdgcn_s_sleep(1);
      }
      if (bad && lane == 0) {
        __builtin_amdgcn_s_sleep(127);
        p.out[rank][(size_t)rank * SH * SH] = (char)-128;
      }
      STAMP(3);
    }
    return;
  }

  const __amdgpu_buffer_rsrc_t rs = RSRC(myws, WS_USED);
  unsigned call;
  {
    const int gw = b * 4 + wid;
    v4u ra[8], rb2[8], o[4];
    const int u0 = gw, u1 = gw + 1024;
    const int k0 = u0 / 13, s0 = u0 - k0 * 13;
    const int k1 = u1 / 13, s1 = u1 - k1 * 13;
    const bool two = u1 < 128 * 13;
    unit_load(ra, rx, rw, s0 >= 5, s0 >= 5 ? s0 - 5 : s0, k0, lane);
    if (two) unit_load(rb2, rx, rw, s1 >= 5, s1 >= 5 ? s1 - 5 : s1, k1, lane);
    call = __hip_atomic_load(cntp, __ATOMIC_RELAXED, __HIP_MEMORY_SCOPE_AGENT) + 1;
    const int d0 = s0 >= 5 ? 32 + (s0 - 5) : jq * 5 + s0;
    const int d1 = s1 >= 5 ? 32 + (s1 - 5) : jq * 5 + s1;
    unit_cvt(o, ra, s0 >= 5);
    unit_store(o, rs, stage_off(d0, k0), lane);
    if (two) {
      unit_cvt(o, rb2, s1 >= 5);
      unit_store(o, rs, stage_off(d1, k1), lane);
    }
    WAITV(0);
    if (lane == 0) {
      __builtin_amdgcn_raw_buffer_store_b32(call, rs, flag_off(d0, k0), 0, AUX_SYS);
      if (two) __builtin_amdgcn_raw_buffer_store_b32(call, rs, flag_off(d1, k1), 0, AUX_SYS);
    }
  }
  STAMP(1);
  const int rb = (xcd >> 1) * 8 + (bi & 7);
  const int cti = bi >> 3;
  const int ct = (cti & 1) + 2 * (xcd & 1) + 4 * (cti >> 1);
  f32x4 acc[4][8];
#pragma unroll
  for (int m = 0; m < 4; ++m)
#pragma unroll
    for (int n = 0; n < 8; ++n) acc[m][n] = f32x4{0.f, 0.f, 0.f, 0.f};

  const int fblk = lane == 0 ? rb : 32 + ct * 2 + (lane == 1 ? 0 : 1);
  const u64* fptr = (const u64*)(myws + FLAG_OFF + fblk * 64 * 128);
  unsigned xo[4];
#pragma unroll
  for (int m = 0; m < 4; ++m) xo[m] = XS_OFF + rb * 8192 + ((m + wid) & 3) * 1024 + lane * 16;
  const unsigned wo = WS_OFF + ct * 2 * 8192 + lane * 16;

#pragma unroll 1
  for (int kc = wid; kc < 64; kc += 4) {
    {
      u64 t0 = 0;
      for (;;) {
        bool ok = true;
        if (lane < 3) {
          const u64 f = __hip_atomic_load(fptr + kc * 16, __ATOMIC_RELAXED, __HIP_MEMORY_SCOPE_SYSTEM);
          ok = (unsigned)f >= call && (unsigned)(f >> 32) >= call;
        }
        if (__all(ok)) break;
        if (spin_fail(t0, tmo, call)) break;
        __builtin_amdgcn_s_sleep(2);
      }
      CBAR();
    }
    v4u xf[2][4], wf[2][8];
#pragma unroll
    for (int h = 0; h < 2; ++h) {
#pragma unroll
      for (int m = 0; m < 4; ++m)
        xf[h][m] = __builtin_amdgcn_raw_buffer_load_b128(rs, xo[m] + h * 4096, kc * (32 * 8192), AUX_SYS);
#pragma unroll
      for (int n = 0; n < 8; ++n)
        wf[h][n] = __builtin_amdgcn_raw_buffer_load_b128(rs, wo + (n >> 2) * 8192 + h * 4096 + (n & 3) * 1024,
                                                         kc * (16 * 8192), AUX_SYS);
    }
#pragma unroll
    for (int h = 0; h < 2; ++h)
#pragma unroll
      for (int n = 0; n < 8; ++n)
#pragma unroll
        for (int m = 0; m < 4; ++m)
          acc[m][n] = __builtin_amdgcn_mfma_f32_16x16x32_bf16(__builtin_bit_cast(bf16x8, wf[h][n]),
                                                              __builtin_bit_cast(bf16x8, xf[h][m]), acc[m][n], 0, 0, 0);
  }
  STAMP(8);
#pragma unroll
  for (int m = 1; m < 4; ++m) {
    char* dstp = smem + (((wid + m) & 3) * 3 + (m - 1)) * 8192 + lane * 16;
#pragma unroll
    for (int n = 0; n < 8; ++n) *(f32x4*)(dstp + n * 1024) = acc[m][n];
  }
  asm volatile("s_waitcnt lgkmcnt(0)" ::: "memory");
  __builtin_amdgcn_s_barrier();
  CBAR();
  float mx = 0.f;
#pragma unroll
  for (int n = 0; n < 8; ++n) {
    const char* srcp = smem + wid * 3 * 8192 + n * 1024 + lane * 16;
    const f32x4 a = *(const f32x4*)(srcp), c = *(const f32x4*)(srcp + 8192), d = *(const f32x4*)(srcp + 16384);
    f32x4 s = (acc[0][n] + a) + (c + d);
#pragma unroll
    for (int e = 0; e < 4; ++e) { s[e] = fmaxf(s[e], 0.f); mx = fmaxf(mx, s[e]); }
    acc[0][n] = s;
  }
#pragma unroll
  for (int s = 32; s > 0; s >>= 1) mx = fmaxf(mx, __shfl_xor(mx, s));
  if (lane == 0) {
    __hip_atomic_store((u64*)(myws + SLOT1_OFF + xcd * 1024) + bi * 4 + wid,
                       ((u64)call << 32) | __builtin_bit_cast(unsigned, mx), __ATOMIC_RELAXED, __HIP_MEMORY_SCOPE_AGENT);
    if (wid == 0) __hip_atomic_store(cntp, call, __ATOMIC_RELAXED, __HIP_MEMORY_SCOPE_AGENT);
  }
  STAMP(9);
  unsigned gm = 0;
  {
    const u64* s2 = (const u64*)(myws + SLOT2_OFF) + lane;
    u64 t0 = 0;
    for (;;) {
      const u64 a = __hip_atomic_load(s2, __ATOMIC_RELAXED, __HIP_MEMORY_SCOPE_SYSTEM);
      gm = (unsigned)a;
      if (__all((unsigned)(a >> 32) >= call)) break;
      if (spin_fail(t0, tmo, call)) break;
      __builtin_amdgcn_s_sleep(1);
    }
#pragma unroll
    for (int s = 32; s > 0; s >>= 1) gm = max(gm, (unsigned)__shfl_xor((int)gm, s));
  }
  STAMP(10);
  const bool bad = __hip_atomic_load(tmo, __ATOMIC_RELAXED, __HIP_MEMORY_SCOPE_AGENT) == call;
  const float amax = __builtin_bit_cast(float, gm);
  const float scale = amax / 127.0f;
  const float inv = scale > 0.f ? 1.0f / scale : 0.f;
  char* tp = smem + wid * 3 * 8192;
#pragma unroll
  for (int cb = 0; cb < 2; ++cb) {
    v4u q;
#pragma unroll
    for (int e = 0; e < 4; ++e) {
      unsigned wv = 0;
#pragma unroll
      for (int i = 0; i < 4; ++i) {
        float f = __builtin_rintf(acc[0][cb * 4 + i][e] * inv);
        f = fminf(f, 127.f);
        wv |= ((unsigned)(int)f & 0xffu) << (8 * i);
      }
      q[e] = bad ? 0x80808080u : wv;
    }
    *(v4u*)(tp + (lane & 15) * 144 + (cb * 4 + (lane >> 4)) * 16) = q;
  }
  asm volatile("s_waitcnt lgkmcnt(0)" ::: "memory");
  int srank, lrow;
  if (rb < 20) { srank = g * 4 + rb / 5; lrow = (rb % 5) * 64; }
  else { srank = (1 - g) * 4 + (rb - 20) / 3; lrow = (5 + (rb - 20) % 3) * 64; }
  const int grow = srank * SH + lrow + wid * 16;
  const bool remote = ct >= 4;
  const int orank = remote ? (1 - g) * 4 + (ct - 4) : rank;
  const int ocol = remote ? jq * 128 : ct * 128;
  const __amdgpu_buffer_rsrc_t ro = RSRC(p.out[orank & 7], NR * SH * SH);
#pragma unroll
  for (int hh = 0; hh < 2; ++hh) {
    const int r = hh * 8 + (lane >> 3), ch = lane & 7;
    const v4u q = *(const v4u*)(tp + r * 144 + ch * 16);
    const unsigned off = (unsigned)(grow + r) * SH + ocol + ch * 16;
    if (remote) __builtin_amdgcn_raw_buffer_store_b128(q, ro, off, 0, AUX_SYS);
    else __builtin_amdgcn_raw_buffer_store_b128(q, ro, off, 0, 0);
  }
  if (remote) {
    WAITV(0);
    if (lane == 0)
      __hip_atomic_store((unsigned*)(p.ws[orank & 7] + DONE_OFF) + (jq * 32 + rb) * 4 + wid, call, __ATOMIC_RELAXED,
                         __HIP_MEMORY_SCOPE_SYSTEM);
  }
  STAMP(11);
}

extern "C" void kernel_launch(int rank, int nranks, ncclComm_t comm, hipStream_t stream, void* const* d_in,
                              const int* in_sizes, int n_in, void* d_out, int out_size, const int* in_dtypes,
                              int out_dtype, void* const* d_out_all, void* const* d_ws_all, size_t ws_bytes) {
  if (nranks != NR || (unsigned)rank >= (unsigned)NR || n_in < 2 || ws_bytes < (size_t)WS_USED ||
      in_sizes[0] != SH * KD || in_sizes[1] != SH * KD || out_size != NR * SH * SH) {
    fprintf(stderr, "unexpected problem shape\n");
    return;
  }
  Params p;
  p.x = (const float*)d_in[0];
  p.w = (const float*)d_in[1];
  for (int i = 0; i < NR; ++i) {
    p.ws[i] = (char*)d_ws_all[i];
    p.out[i] = (char*)d_out_all[i];
  }
  p.rank = rank;
  HIP_CHECK(hipFuncSetAttribute((const void*)ag_gemm_kernel, hipFuncAttributeMaxDynamicSharedMemorySize, LDS_BYTES));
  ag_gemm_kernel<<<NWG, NTHR, LDS_BYTES, stream>>>(p);
  HIP_CHECK(hipGetLastError());
}
```

```cpp
#include <hip/hip_runtime.h>
#include <rccl/rccl.h>
#include <cstdio>

#define HIP_CHECK(expr) do { hipError_t _e = (expr); if (_e != hipSuccess) { \
    fprintf(stderr, "HIP error %s at %s:%d\n", hipGetErrorString(_e), __FILE__, __LINE__); \
    return; } } while (0)

typedef unsigned v4u __attribute__((ext_vector_type(4)));
typedef unsigned v2u __attribute__((ext_vector_type(2)));
typedef float f32x4 __attribute__((ext_vector_type(4)));
typedef float f32x2 __attribute__((ext_vector_type(2)));
typedef __bf16 bf16x2 __attribute__((ext_vector_type(2)));
typedef short bf16x8 __attribute__((ext_vector_type(8)));
typedef unsigned long long u64;
#define LAS __attribute__((address_space(3)))
#define GAS __attribute__((address_space(1)))

constexpr int NR = 8;
constexpr int KD = 4096;
constexpr int SH = 512;
constexpr int NWG = 256;
constexpr int NTHR = 512;
#define LINK_WGS 14
#define LINK_WAVES 1
constexpr int NLW = LINK_WGS * LINK_WAVES;
constexpr int NSELF = (7 * (16 - LINK_WGS) + 16) * 8;
static_assert(LINK_WGS < 16 && 3 * NLW <= 128 * 5 && 3 * NSELF <= 128 * 13, "three fixed units per wave");
constexpr unsigned XS_OFF = 0;
constexpr unsigned WS_OFF = 16u << 20;
constexpr unsigned FLAG_OFF = 24u << 20;
constexpr unsigned FLAG_PITCH = 4096 + 512;
constexpr unsigned SLOT1_OFF = FLAG_OFF + (256u << 10);
constexpr unsigned SLOT1_PITCH = 1024 + 256;
constexpr unsigned SLOT2_OFF = SLOT1_OFF + (16u << 10);
constexpr unsigned DONE_OFF = SLOT2_OFF + (32u << 10);
constexpr unsigned CNT_OFF = DONE_OFF + (4u << 10);
constexpr unsigned TICK_OFF = CNT_OFF + (16u << 10);
constexpr unsigned TMO_OFF = TICK_OFF + (4u << 10);
constexpr unsigned WS_USED = TMO_OFF + 4096;
constexpr unsigned STAMP_OFF = 32u << 20;
constexpr u64 DEADLINE_TICKS = 50000000ull;
constexpr int CTL = 128 * 1024;
constexpr int LDS_BYTES = CTL + 256;
constexpr unsigned BAD_MAX = 0xfffffffeu;
#define AUX_DMA 17
constexpr int AUX_SYS = 17;

struct Params {
  const float* x;
  const float* w;
  char* ws[NR];
  char* out[NR];
  int rank;
};

#define RSRC(p, bytes) __builtin_amdgcn_make_buffer_rsrc((void*)(p), 0, (int)(bytes), 0x00020000)
#define CBAR() asm volatile("" ::: "memory")
#define WAITV(n) asm volatile("s_waitcnt vmcnt(" #n ")" ::: "memory")
#define WAITL() asm volatile("s_waitcnt lgkmcnt(0)" ::: "memory")
#define RFL(v) __builtin_amdgcn_readfirstlane(v)

#define STAMP(i) do {} while (0)
#define STAMP_END(cls) do {} while (0)

__device__ __forceinline__ unsigned pk(float a, float b) {
  f32x2 v = {a, b};
  return __builtin_bit_cast(unsigned, __builtin_convertvector(v, bf16x2));
}

__device__ __forceinline__ void unit_load(v4u (&r)[8], __amdgpu_buffer_rsrc_t rx, __amdgpu_buffer_rsrc_t rw,
                                          bool isw, int blk, int k32, int lane, unsigned oob) {
  const int kg = lane >> 4, c = lane & 15;
  if (!isw) {
    const unsigned base = ((unsigned)(blk * 64 + c) * (KD * 4) + (unsigned)(k32 * 32 + 4 * kg) * 4) | oob;
#pragma unroll
    for (int m = 0; m < 4; ++m) {
      r[2 * m] = __builtin_amdgcn_raw_buffer_load_b128(rx, base + m * (16 * KD * 4), 0, 0);
      r[2 * m + 1] = __builtin_amdgcn_raw_buffer_load_b128(rx, base + m * (16 * KD * 4) + 64, 0, 0);
    }
  } else {
    const unsigned base = ((unsigned)(k32 * 32 + 4 * kg) * (SH * 4) + (unsigned)(blk * 64 + 4 * c) * 4) | oob;
#pragma unroll
    for (int jj = 0; jj < 8; ++jj)
      r[jj] = __builtin_amdgcn_raw_buffer_load_b128(rw, base + ((jj & 3) + 16 * (jj >> 2)) * (SH * 4), 0, 0);
  }
}
__device__ __forceinline__ float asf(unsigned u) { return __builtin_bit_cast(float, u); }
__device__ __forceinline__ void unit_cvt(v4u (&o)[4], const v4u (&r)[8], bool isw) {
  if (!isw) {
#pragma unroll
    for (int m = 0; m < 4; ++m) {
      o[m].x = pk(asf(r[2 * m].x), asf(r[2 * m].y));
      o[m].y = pk(asf(r[2 * m].z), asf(r[2 * m].w));
      o[m].z = pk(asf(r[2 * m + 1].x), asf(r[2 * m + 1].y));
      o[m].w = pk(asf(r[2 * m + 1].z), asf(r[2 * m + 1].w));
    }
  } else {
#pragma unroll
    for (int i = 0; i < 4; ++i) {
      o[i].x = pk(asf(r[0][i]), asf(r[1][i]));
      o[i].y = pk(asf(r[2][i]), asf(r[3][i]));
      o[i].z = pk(asf(r[4][i]), asf(r[5][i]));
      o[i].w = pk(asf(r[6][i]), asf(r[7][i]));
    }
  }
}
__device__ __forceinline__ unsigned stage_off(int dblk, int k32) {
  const int kc = k32 >> 1, h = k32 & 1;
  return dblk < 32 ? XS_OFF + (unsigned)((kc * 32 + dblk) * 2 + h) * 4096u
                   : WS_OFF + (unsigned)((kc * 16 + (dblk - 32)) * 2 + h) * 4096u;
}
__device__ __forceinline__ unsigned flag_off(int dblk, int k32) {
  return FLAG_OFF + (unsigned)dblk * FLAG_PITCH + (unsigned)k32 * 4u;
}
__device__ __forceinline__ void unit_store(const v4u (&o)[4], __amdgpu_buffer_rsrc_t rd, unsigned off, int lane) {
#pragma unroll
  for (int i = 0; i < 4; ++i) __builtin_amdgcn_raw_buffer_store_b128(o[i], rd, off + lane * 16 + i * 1024, 0, AUX_SYS);
}

__device__ __forceinline__ bool spin_fail(u64& t0, unsigned* tmo, unsigned call) {
  const u64 now = __builtin_amdgcn_s_memrealtime();
  if (t0 == 0) t0 = now;
  if (now - t0 > 10000) {
    if (__hip_atomic_load(tmo, __ATOMIC_RELAXED, __HIP_MEMORY_SCOPE_AGENT) == call) return true;
    if (now - t0 > DEADLINE_TICKS) {
      __hip_atomic_store(tmo, call, __ATOMIC_RELAXED, __HIP_MEMORY_SCOPE_AGENT);
      return true;
    }
  }
  return false;
}

__device__ __forceinline__ unsigned wave_umax(unsigned v) {
  v = max(v, (unsigned)__builtin_amdgcn_update_dpp(0, (int)v, 0x111, 0xf, 0xf, true));
  v = max(v, (unsigned)__builtin_amdgcn_update_dpp(0, (int)v, 0x112, 0xf, 0xf, true));
  v = max(v, (unsigned)__builtin_amdgcn_update_dpp(0, (int)v, 0x114, 0xf, 0xf, true));
  v = max(v, (unsigned)__builtin_amdgcn_update_dpp(0, (int)v, 0x118, 0xf, 0xf, true));
  const unsigned a = __builtin_amdgcn_readlane(v, 15), b = __builtin_amdgcn_readlane(v, 31);
  const unsigned c = __builtin_amdgcn_readlane(v, 47), d = __builtin_amdgcn_readlane(v, 63);
  return max(max(a, b), max(c, d));
}

__global__ void __launch_bounds__(NTHR, 2) ag_gemm_kernel(Params p) {
  extern __shared__ __attribute__((aligned(16))) char smem[];
  LAS char* const L8 = (LAS char*)smem;
  const int lane = threadIdx.x & 63;
  const int wid = RFL(threadIdx.x >> 6);
  const int b = blockIdx.x;
  const int rank = p.rank, g = rank >> 2, jq = rank & 3;
  const int xcd = b & 7, bi = b >> 3;
  char* const myws = p.ws[rank];
  unsigned* const tmo = (unsigned*)(myws + TMO_OFF);
  unsigned* const cntp = (unsigned*)(myws + CNT_OFF + b * 64 + wid * 4);
  STAMP(0);

  if (bi < 16) {
    const bool link = xcd < 7 && bi < LINK_WGS;
    if (link && wid >= LINK_WAVES) { STAMP_END(0); return; }
    const __amdgpu_buffer_rsrc_t rx = RSRC(p.x, SH * KD * 4);
    const __amdgpu_buffer_rsrc_t rw = RSRC(p.w, SH * KD * 4);
    const int dst = link ? (rank + 1 + xcd) & 7 : rank;
    const int mode = !link ? 2 : ((dst >> 2) == g ? 0 : 1);
    const int step = link ? NLW : NSELF;
    const int total = link ? 128 * 5 : 128 * 13;
    int u;
    if (link) u = wid * LINK_WGS + bi;
    else u = wid * (NSELF / 8) + (xcd == 7 ? bi : 16 + xcd * (16 - LINK_WGS) + (bi - LINK_WGS));
    const __amdgpu_buffer_rsrc_t rd = RSRC(p.ws[dst], WS_USED);
    auto decode = [&](int uu, bool& isw, int& sblk, int& dblk, int& k32) {
      if (mode == 2) {
        k32 = uu / 13;
        const int s = uu - k32 * 13;
        isw = s >= 5; sblk = isw ? s - 5 : s; dblk = isw ? 32 + s - 5 : jq * 5 + s;
      } else {
        k32 = uu / 5;
        const int s = uu - k32 * 5;
        if (mode == 0) { isw = false; sblk = s; dblk = jq * 5 + s; }
        else if (s < 3) { isw = false; sblk = 5 + s; dblk = 20 + jq * 3 + s; }
        else { isw = true; sblk = (dst & 3) * 2 + (s - 3); dblk = 32 + 8 + jq * 2 + (s - 3); }
      }
    };
    v4u ra[8], rb2[8], o[4];
    bool isw, isw2; int sblk, dblk, k32, sblk2, dblk2, k322;
    decode(u, isw, sblk, dblk, k32);
    unit_load(ra, rx, rw, isw, sblk, k32, lane, 0);
    unsigned call;
    asm volatile("global_load_dword %0, %1, %2 sc1" : "=v"(call) : "v"(0), "s"(cntp) : "memory");
    unsigned pflag = 0xffffffffu;
    const unsigned lmask = lane == 0 ? 0u : 0xffffffffu;
    const __amdgpu_buffer_rsrc_t rs = RSRC(myws, WS_USED);
    const unsigned tko = TICK_OFF + (link ? xcd : 7) * 256;
    unsigned tk = 1, tbase = 0;
    int stat = u + 2 * step;
    u += step;
    int t = 0;
#define DRAW() do { u64 sv_; tk = 1; asm volatile("s_mov_b64 %1, exec\n\ts_cmp_lt_u32 %4, %5\n\ts_cselect_b64 exec, 1, 0\n\tbuffer_atomic_add %0, %2, %3, 0 offen sc0 sc1\n\ts_mov_b64 exec, %1" : "+v"(tk), "=&s"(sv_) : "v"(tko), "s"(rs), "s"(u), "s"(total) : "memory", "scc"); } while (0)
#pragma unroll 1
    for (;;) {
      bool more = (unsigned)u < (unsigned)total;
      decode(u, isw2, sblk2, dblk2, k322);
      unit_load(rb2, rx, rw, isw2, sblk2, k322, lane, more ? 0u : 0x80000000u);
      WAITV(8);
      asm volatile("" : "+v"(call), "+v"(tk));
      tbase = (unsigned)RFL(call) * (unsigned)(total - 2 * step) - 3u * step;
      u = stat >= 0 ? stat : (int)((unsigned)RFL(tk) - tbase);
      stat = -1;
      STAMP(1 + t);
      __builtin_amdgcn_raw_buffer_store_b32(call + 1, rd, pflag | lmask, 0, AUX_SYS);
      unit_cvt(o, ra, isw);
      unit_store(o, rd, stage_off(dblk, k32), lane);
      pflag = flag_off(dblk, k32);
      if (!more) break;
      more = (unsigned)u < (unsigned)total;
      DRAW();
      decode(u, isw, sblk, dblk, k32);
      unit_load(ra, rx, rw, isw, sblk, k32, lane, more ? 0u : 0x80000000u);
      WAITV(8);
      asm volatile("" : "+v"(tk));
      u = (int)((unsigned)RFL(tk) - tbase);
      STAMP(2 + t);
      __builtin_amdgcn_raw_buffer_store_b32(call + 1, rd, pflag | lmask, 0, AUX_SYS);
      unit_cvt(o, rb2, isw2);
      unit_store(o, rd, stage_off(dblk2, k322), lane);
      pflag = flag_off(dblk2, k322);
      if (!more) break;
      DRAW();
      t = min(t + 2, 56);
    }
    call += 1;
    WAITV(0);
    STAMP(60);
    if (lane == 0) {
      __builtin_amdgcn_raw_buffer_store_b32(call, rd, pflag, 0, AUX_SYS);
      __hip_atomic_store(cntp, call, __ATOMIC_RELAXED, __HIP_MEMORY_SCOPE_AGENT);
    }
    if (bi == 15 && (wid < 2 || (b == 127 && wid >= 4))) {
      u64 t0 = 0;
      if (wid < 2) {
        const __amdgpu_buffer_rsrc_t rq = RSRC(p.ws[xcd], WS_USED);
        unsigned m;
        for (;;) {
          const v4u a = __builtin_amdgcn_raw_buffer_load_b128(rs, SLOT1_OFF + xcd * SLOT1_PITCH + lane * 16, 0, 16);
          CBAR();
          m = max(a.x, a.z);
          if (__all(a.y == call && a.w == call)) break;
          if (spin_fail(t0, tmo, call)) { m = BAD_MAX; break; }
        }
        const v2u mv = {wave_umax(m), call};
        __builtin_amdgcn_raw_buffer_store_b64(mv, rq, SLOT2_OFF + lane * 256 + rank * 8, 0, AUX_SYS);
        __builtin_amdgcn_raw_buffer_store_b64(mv, rq, SLOT2_OFF + (lane + 64) * 256 + rank * 8, 0, AUX_SYS);
        STAMP(61);
      } else {
        bool bad = false;
        for (;;) {
          const unsigned a = __builtin_amdgcn_raw_buffer_load_b32(rs, DONE_OFF + (wid - 4) * 256 + lane * 4, 0, AUX_SYS);
          CBAR();
          if (__all(a >= call)) break;
          if (spin_fail(t0, tmo, call)) { bad = true; break; }
        }
        if (bad && lane == 0) {
          __builtin_amdgcn_s_sleep(127);
          p.out[rank][(size_t)rank * SH * SH] = (char)-128;
        }
        STAMP(62);
      }
      STAMP_END(3);
      return;
    }
    STAMP_END(link ? 2 : 5);
    return;
  }

  volatile LAS unsigned* const ctl = (volatile LAS unsigned*)(L8 + CTL);
  const unsigned call = __hip_atomic_load(cntp, __ATOMIC_RELAXED, __HIP_MEMORY_SCOPE_AGENT) + 1;
  if (threadIdx.x < 16) ctl[threadIdx.x] = 0;
  if (lane == 0) __hip_atomic_store(cntp, call, __ATOMIC_RELAXED, __HIP_MEMORY_SCOPE_AGENT);
  WAITL();
  __builtin_amdgcn_s_barrier();
  CBAR();
  const int ti = bi - 16;
  const int rbb = (xcd >> 1) * 4 + (ti & 3);
  const int cti = ti >> 2;
  const int ct = (cti & 1) + 2 * (xcd & 1) + 4 * (cti >> 1);
  const __amdgpu_buffer_rsrc_t rs = RSRC(myws, WS_USED);

  if (wid >= 4) {
    const int s = wid - 4;
    const int blk = s < 2 ? 2 * rbb + s : 32 + 2 * ct + (s - 2);
    const unsigned sbase = (s < 2 ? XS_OFF + blk * 8192 : WS_OFF + (blk - 32) * 8192) + lane * 16;
    const unsigned sstep = s < 2 ? 32 * 8192 : 16 * 8192;
    const int c0 = s < 2 ? 2 * s : s - 2, c1 = s < 2 ? 2 * s + 1 : s;
    const unsigned fo = FLAG_OFF + blk * FLAG_PITCH + lane * 8;
    int avail = 0, issued = 0, pub = 0;
    u64 t0 = 0;
#pragma unroll 1
    while (pub < 64) {
      const int fr = RFL(min(ctl[4 + c0], ctl[4 + c1])) + 4;
      const int lim = min(avail, fr);
      CBAR();
#pragma unroll 1
      while (issued < lim) {
        LAS char* dl = L8 + s * 32768 + (issued & 3) * 8192;
#pragma unroll
        for (int i = 0; i < 8; ++i)
          __builtin_amdgcn_raw_ptr_buffer_load_lds(rs, (LAS void*)(dl + i * 1024), 16, sbase + i * 1024, issued * sstep, 0, AUX_DMA);
        ++issued;
      }
      if (issued == avail && avail < 64) {
        v2u f = {0u, 0u};
        const bool inw = (unsigned)(lane - avail) < 16u;
        if (inw) f = __builtin_amdgcn_raw_buffer_load_b64(rs, fo, 0, AUX_SYS);
        const int o = issued - pub;
        if (o >= 4) { WAITV(25); ctl[s] = issued - 3; }
        if (o >= 3) { WAITV(17); ctl[s] = issued - 2; }
        if (o >= 2) { WAITV(9); ctl[s] = issued - 1; }
        if (o >= 1) { WAITV(1); ctl[s] = issued; STAMP(1 + (issued >> 1)); }
        pub = issued;
        CBAR();
        const u64 mk = __ballot(lane < avail || (inw && f.x >= call && f.y >= call));
        const int na = ~mk ? __builtin_ctzll(~mk) : 64;
        if (na == avail) {
          if (spin_fail(t0, tmo, call)) { avail = 64; ctl[12] = 1; }
        } else {
          avail = na; t0 = 0;
        }
      } else if (pub < issued) {
        const int o = issued - pub;
        if (o >= 4) WAITV(24);
        else if (o == 3) WAITV(16);
        else if (o == 2) WAITV(8);
        else WAITV(0);
        ++pub;
        ctl[s] = pub;
        STAMP(1 + (pub >> 1));
        t0 = 0;
      } else {
        if (spin_fail(t0, tmo, call)) { ctl[12] = 1; break; }
      }
    }
    WAITV(0);
    WAITL();
    STAMP_END(4);
    return;
  }

  const int wm = wid >> 1, wn = wid & 1;
  const bool remote = ct >= 4;
  const int orank = (remote ? (1 - g) * 4 + (ct - 4) : rank) & 7;
  u64 donep = (u64)p.ws[orank] + DONE_OFF + ((jq * 16 + rbb) * 4 + wid) * 4;
  unsigned outlo = (unsigned)(u64)p.out[orank], outhi = (unsigned)((u64)p.out[orank] >> 32);
  asm volatile("" : "+v"(donep), "+s"(outlo), "+s"(outhi));
  char* const outp = (char*)(((u64)(unsigned)RFL(outhi) << 32) | (u64)(unsigned)RFL(outlo));
  f32x4 acc[4][4];
#pragma unroll
  for (int m = 0; m < 4; ++m)
#pragma unroll
    for (int n = 0; n < 4; ++n) acc[m][n] = f32x4{0.f, 0.f, 0.f, 0.f};
  {
    const LAS char* xb = L8 + wm * 32768 + lane * 16;
    const LAS char* wb = L8 + (2 + wn) * 32768 + lane * 16;
    int full = 0;
    v4u xa[4], wa[4], xc[4], wc[4];
    auto rd_half = [&](v4u (&xf)[4], v4u (&wf)[4], int off) {
#pragma unroll
      for (int m = 0; m < 4; ++m) wf[m] = *(const LAS v4u*)(wb + off + m * 1024);
#pragma unroll
      for (int m = 0; m < 4; ++m) xf[m] = *(const LAS v4u*)(xb + off + m * 1024);
    };
    auto mm = [&](const v4u (&xf)[4], const v4u (&wf)[4]) {
#pragma unroll
      for (int m = 0; m < 4; ++m)
#pragma unroll
        for (int n = 0; n < 4; ++n)
          acc[m][n] = __builtin_amdgcn_mfma_f32_16x16x32_bf16(__builtin_bit_cast(bf16x8, wf[n]),
                                                              __builtin_bit_cast(bf16x8, xf[m]), acc[m][n], 0, 0, 0);
    };
    auto wait_full = [&](int kc) {
      u64 t0 = 0;
      int sp = 0;
      while (full <= kc) {
        full = RFL(min(ctl[wm], ctl[2 + wn]));
        if ((++sp & 63) == 0 && spin_fail(t0, tmo, call)) { full = 64; ctl[12] = 1; }
      }
      CBAR();
    };
    wait_full(0);
    rd_half(xa, wa, 0);
#pragma unroll 1
    for (int kc = 0; kc < 64; ++kc) {
      if ((kc & 1) == 0) STAMP(1 + (kc >> 1));
      rd_half(xc, wc, (kc & 3) * 8192 + 4096);
      const unsigned f0 = ctl[wm], f1 = ctl[2 + wn];
      __builtin_amdgcn_sched_barrier(0);
      mm(xa, wa);
      __builtin_amdgcn_sched_barrier(0);
      WAITL();
      ctl[4 + wid] = kc + 1;
      full = max(full, RFL(min(f0, f1)));
      const bool ahead = full > kc + 1;
      if (ahead) rd_half(xa, wa, ((kc + 1) & 3) * 8192);
      __builtin_amdgcn_sched_barrier(0);
      mm(xc, wc);
      __builtin_amdgcn_sched_barrier(0);
      if (!ahead && kc < 63) {
        wait_full(kc + 1);
        rd_half(xa, wa, ((kc + 1) & 3) * 8192);
      }
    }
  }
  STAMP(34);
  float mx = 0.f;
#pragma unroll
  for (int m = 0; m < 4; ++m)
#pragma unroll
    for (int n = 0; n < 4; ++n)
#pragma unroll
      for (int e = 0; e < 4; ++e) mx = fmaxf(mx, acc[m][n][e]);
  bool bad;
  {
    const unsigned wmx = wave_umax(__builtin_bit_cast(unsigned, mx));
    if (lane == 0) ctl[8 + wid] = wmx;
    WAITL();
    __builtin_amdgcn_s_barrier();
    CBAR();
    if (wid == 0 && lane < 8) {
      const unsigned m4 = max(max(ctl[8], ctl[9]), max(ctl[10], ctl[11]));
      const v2u mv = {m4, call};
      __builtin_amdgcn_raw_buffer_store_b64(mv, rs, SLOT1_OFF + lane * SLOT1_PITCH + (b - 128) * 8, 0, 16);
    }
    bad = ctl[12] != 0;
  }
  STAMP(35);
#pragma unroll
  for (int m = 0; m < 4; ++m)
#pragma unroll
    for (int n = 0; n < 4; ++n)
#pragma unroll
      for (int e = 0; e < 4; ++e) acc[m][n][e] = fmaxf(acc[m][n][e], 0.f);
#pragma unroll
  for (int m = 0; m < 4; ++m) asm volatile("" : "+v"(acc[m][0]), "+v"(acc[m][1]), "+v"(acc[m][2]), "+v"(acc[m][3]));
  unsigned gm;
  {
    const unsigned so = SLOT2_OFF + (b - 128) * 256 + (lane & 7) * 8;
    u64 t0 = 0;
    for (int sp = 1;; ++sp) {
      if (wid < 2) {
        const v2u a = __builtin_amdgcn_raw_buffer_load_b64(rs, so, 0, AUX_SYS);
        CBAR();
        if (__all(a.y >= call)) {
          gm = a.x;
          gm = max(gm, (unsigned)__builtin_amdgcn_update_dpp(0, (int)gm, 0x111, 0xf, 0xf, true));
          gm = max(gm, (unsigned)__builtin_amdgcn_update_dpp(0, (int)gm, 0x112, 0xf, 0xf, true));
          gm = max(gm, (unsigned)__builtin_amdgcn_update_dpp(0, (int)gm, 0x114, 0xf, 0xf, true));
          gm = __builtin_amdgcn_readlane(gm, 7);
          ctl[13] = gm + 1;
          break;
        }
      }
      gm = RFL(ctl[13]) - 1;
      if (gm != 0xffffffffu) break;
      if ((wid < 2 || (sp & 63) == 0) && spin_fail(t0, tmo, call)) { gm = BAD_MAX; break; }
    }
  }
  STAMP(36);
  bad = bad || gm == BAD_MAX;
  const float amax = __builtin_bit_cast(float, gm);
  const float inv = amax > 0.f && !bad ? 127.0f * __builtin_amdgcn_rcpf(amax) : 0.f;
  const float magic = bad ? 12582912.0f + 128.0f : 12582912.0f;
  const int rb = 2 * rbb + wm;
  int srank, lrow;
  if (rb < 20) { srank = g * 4 + rb / 5; lrow = (rb % 5) * 64; }
  else { srank = (1 - g) * 4 + (rb - 20) / 3; lrow = (5 + (rb - 20) % 3) * 64; }
  const int ocol = (remote ? jq * 128 : ct * 128) + wn * 64;
  const __amdgpu_buffer_rsrc_t ro = RSRC(outp, NR * SH * SH);
  const unsigned obase = (unsigned)(srank * SH + lrow + (lane & 15)) * SH + ocol + (lane >> 4) * 16;
#pragma unroll
  for (int m = 0; m < 4; ++m) {
    v4u q;
    const f32x4 inv4 = {inv, inv, inv, inv}, mg4 = {magic, magic, magic, magic};
    v4u tb[4];
#pragma unroll
    for (int n = 0; n < 4; ++n) tb[n] = __builtin_bit_cast(v4u, __builtin_elementwise_fma(acc[m][n], inv4, mg4));
#pragma unroll
    for (int e = 0; e < 4; ++e) {
      const unsigned lo = __builtin_amdgcn_perm(tb[1][e], tb[0][e], 0x0c0c0400u);
      const unsigned hi = __builtin_amdgcn_perm(tb[3][e], tb[2][e], 0x04000c0cu);
      q[e] = lo | hi;
    }
    __builtin_amdgcn_raw_buffer_store_b128(q, ro, obase + m * (16 * SH), 0, AUX_SYS);
  }
  STAMP(37);
  if (remote) {
    WAITV(0);
    if (lane == 0)
      __hip_atomic_store((GAS unsigned*)donep, call, __ATOMIC_RELAXED, __HIP_MEMORY_SCOPE_SYSTEM);
  }
  STAMP(38);
  STAMP_END(1);
}

extern "C" void kernel_launch(int rank, int nranks, ncclComm_t comm, hipStream_t stream, void* const* d_in,
                              const int* in_sizes, int n_in, void* d_out, int out_size, const int* in_dtypes,
                              int out_dtype, void* const* d_out_all, void* const* d_ws_all, size_t ws_bytes) {
  if (nranks != NR || (unsigned)rank >= (unsigned)NR || n_in < 2 || ws_bytes < (size_t)WS_USED ||
      in_sizes[0] != SH * KD || in_sizes[1] != SH * KD || out_size != NR * SH * SH) {
    fprintf(stderr, "unexpected problem shape\n");
    return;
  }
  Params p;
  p.x = (const float*)d_in[0];
  p.w = (const float*)d_in[1];
  for (int i = 0; i < NR; ++i) {
    p.ws[i] = (char*)d_ws_all[i];
    p.out[i] = (char*)d_out_all[i];
  }
  p.rank = rank;
  HIP_CHECK(hipFuncSetAttribute((const void*)ag_gemm_kernel, hipFuncAttributeMaxDynamicSharedMemorySize, LDS_BYTES));
  ag_gemm_kernel<<<NWG, NTHR, LDS_BYTES, stream>>>(p);
  HIP_CHECK(hipGetLastError());
}
```

```cpp
#include <hip/hip_runtime.h>
#include <rccl/rccl.h>
#include <cstdio>

#define HIP_CHECK(expr) do { hipError_t _e = (expr); if (_e != hipSuccess) { \
    fprintf(stderr, "HIP error %s at %s:%d\n", hipGetErrorString(_e), __FILE__, __LINE__); \
    return; } } while (0)

typedef unsigned v4u __attribute__((ext_vector_type(4)));
typedef unsigned v2u __attribute__((ext_vector_type(2)));
typedef float f32x4 __attribute__((ext_vector_type(4)));
typedef float f32x2 __attribute__((ext_vector_type(2)));
typedef __bf16 bf16x2 __attribute__((ext_vector_type(2)));
typedef short bf16x8 __attribute__((ext_vector_type(8)));
typedef unsigned long long u64;
#define LAS __attribute__((address_space(3)))
#define GAS __attribute__((address_space(1)))

constexpr int NR = 8;
constexpr int KD = 4096;
constexpr int SH = 512;
constexpr int NWG = 256;
constexpr int NTHR = 512;
#define LINK_WGS 14
#define LINK_WAVES 1
constexpr unsigned PHYS = 0x75416320u;
__host__ __device__ constexpr int phys(int r) { return (PHYS >> (4 * r)) & 7; }
constexpr int NLW = LINK_WGS * LINK_WAVES;
constexpr int NSELF = (7 * (16 - LINK_WGS) + 16) * 8;
static_assert(LINK_WGS < 16 && 3 * NLW <= 128 * 5 && 3 * NSELF <= 128 * 13, "three fixed units per wave");
constexpr unsigned XS_OFF = 0;
constexpr unsigned WS_OFF = 16u << 20;
constexpr unsigned FLAG_OFF = 24u << 20;
constexpr unsigned FLAG_PITCH = 4096 + 512;
constexpr unsigned SLOT1_OFF = FLAG_OFF + (256u << 10);
constexpr unsigned SLOT1_PITCH = 1024 + 256;
constexpr unsigned SLOT2_OFF = SLOT1_OFF + (16u << 10);
constexpr unsigned DONE_OFF = SLOT2_OFF + (32u << 10);
constexpr unsigned CNT_OFF = DONE_OFF + (4u << 10);
constexpr unsigned TICK_OFF = CNT_OFF + (16u << 10);
constexpr unsigned TMO_OFF = TICK_OFF + (4u << 10);
constexpr unsigned WS_USED = TMO_OFF + 4096;
constexpr unsigned STAMP_OFF = 32u << 20;
constexpr u64 DEADLINE_TICKS = 50000000ull;
constexpr int CTL = 128 * 1024;
constexpr int LDS_BYTES = CTL + 256;
constexpr unsigned BAD_MAX = 0xfffffffeu;
#define AUX_DMA 17
constexpr int AUX_SYS = 17;

struct Params {
  const float* x;
  const float* w;
  char* ws[NR];
  char* out[NR];
  int rank;
};

#define RSRC(p, bytes) __builtin_amdgcn_make_buffer_rsrc((void*)(p), 0, (int)(bytes), 0x00020000)
#define CBAR() asm volatile("" ::: "memory")
#define WAITV(n) asm volatile("s_waitcnt vmcnt(" #n ")" ::: "memory")
#define WAITL() asm volatile("s_waitcnt lgkmcnt(0)" ::: "memory")
#define RFL(v) __builtin_amdgcn_readfirstlane(v)

#define STAMP(i) do {} while (0)
#define STAMP_END(cls) do {} while (0)

__device__ __forceinline__ unsigned pk(float a, float b) {
  f32x2 v = {a, b};
  return __builtin_bit_cast(unsigned, __builtin_convertvector(v, bf16x2));
}

__device__ __forceinline__ void unit_load(v4u (&r)[8], __amdgpu_buffer_rsrc_t rx, __amdgpu_buffer_rsrc_t rw,
                                          bool isw, int blk, int k32, int lane, unsigned oob) {
  const int kg = lane >> 4, c = lane & 15;
  if (!isw) {
    const unsigned base = ((unsigned)(blk * 64 + c) * (KD * 4) + (unsigned)(k32 * 32 + 4 * kg) * 4) | oob;
#pragma unroll
    for (int m = 0; m < 4; ++m) {
      r[2 * m] = __builtin_amdgcn_raw_buffer_load_b128(rx, base + m * (16 * KD * 4), 0, 0);
      r[2 * m + 1] = __builtin_amdgcn_raw_buffer_load_b128(rx, base + m * (16 * KD * 4) + 64, 0, 0);
    }
  } else {
    const unsigned base = ((unsigned)(k32 * 32 + 4 * kg) * (SH * 4) + (unsigned)(blk * 64 + 4 * c) * 4) | oob;
#pragma unroll
    for (int jj = 0; jj < 8; ++jj)
      r[jj] = __builtin_amdgcn_raw_buffer_load_b128(rw, base + ((jj & 3) + 16 * (jj >> 2)) * (SH * 4), 0, 0);
  }
}
__device__ __forceinline__ float asf(unsigned u) { return __builtin_bit_cast(float, u); }
__device__ __forceinline__ void unit_cvt(v4u (&o)[4], const v4u (&r)[8], bool isw) {
  if (!isw) {
#pragma unroll
    for (int m = 0; m < 4; ++m) {
      o[m].x = pk(asf(r[2 * m].x), asf(r[2 * m].y));
      o[m].y = pk(asf(r[2 * m].z), asf(r[2 * m].w));
      o[m].z = pk(asf(r[2 * m + 1].x), asf(r[2 * m + 1].y));
      o[m].w = pk(asf(r[2 * m + 1].z), asf(r[2 * m + 1].w));
    }
  } else {
#pragma unroll
    for (int i = 0; i < 4; ++i) {
      o[i].x = pk(asf(r[0][i]), asf(r[1][i]));
      o[i].y = pk(asf(r[2][i]), asf(r[3][i]));
      o[i].z = pk(asf(r[4][i]), asf(r[5][i]));
      o[i].w = pk(asf(r[6][i]), asf(r[7][i]));
    }
  }
}
__device__ __forceinline__ unsigned stage_off(int dblk, int k32) {
  const int kc = k32 >> 1, h = k32 & 1;
  return dblk < 32 ? XS_OFF + (unsigned)((kc * 32 + dblk) * 2 + h) * 4096u
                   : WS_OFF + (unsigned)((kc * 16 + (dblk - 32)) * 2 + h) * 4096u;
}
__device__ __forceinline__ unsigned flag_off(int dblk, int k32) {
  return FLAG_OFF + (unsigned)dblk * FLAG_PITCH + (unsigned)k32 * 4u;
}
__device__ __forceinline__ void unit_store(const v4u (&o)[4], __amdgpu_buffer_rsrc_t rd, unsigned off, int lane) {
#pragma unroll
  for (int i = 0; i < 4; ++i) __builtin_amdgcn_raw_buffer_store_b128(o[i], rd, off + lane * 16 + i * 1024, 0, AUX_SYS);
}

__device__ __forceinline__ bool spin_fail(u64& t0, unsigned* tmo, unsigned call) {
  const u64 now = __builtin_amdgcn_s_memrealtime();
  if (t0 == 0) t0 = now;
  if (now - t0 > 10000) {
    if (__hip_atomic_load(tmo, __ATOMIC_RELAXED, __HIP_MEMORY_SCOPE_AGENT) == call) return true;
    if (now - t0 > DEADLINE_TICKS) {
      __hip_atomic_store(tmo, call, __ATOMIC_RELAXED, __HIP_MEMORY_SCOPE_AGENT);
      return true;
    }
  }
  return false;
}

__device__ __forceinline__ unsigned wave_umax(unsigned v) {
  v = max(v, (unsigned)__builtin_amdgcn_update_dpp(0, (int)v, 0x111, 0xf, 0xf, true));
  v = max(v, (unsigned)__builtin_amdgcn_update_dpp(0, (int)v, 0x112, 0xf, 0xf, true));
  v = max(v, (unsigned)__builtin_amdgcn_update_dpp(0, (int)v, 0x114, 0xf, 0xf, true));
  v = max(v, (unsigned)__builtin_amdgcn_update_dpp(0, (int)v, 0x118, 0xf, 0xf, true));
  const unsigned a = __builtin_amdgcn_readlane(v, 15), b = __builtin_amdgcn_readlane(v, 31);
  const unsigned c = __builtin_amdgcn_readlane(v, 47), d = __builtin_amdgcn_readlane(v, 63);
  return max(max(a, b), max(c, d));
}

__global__ void __launch_bounds__(NTHR, 2) ag_gemm_kernel(Params p) {
  extern __shared__ __attribute__((aligned(16))) char smem[];
  LAS char* const L8 = (LAS char*)smem;
  const int lane = threadIdx.x & 63;
  const int wid = RFL(threadIdx.x >> 6);
  const int b = blockIdx.x;
  const int rank = p.rank, g = rank >> 2, jq = rank & 3;
  const int xcd = b & 7, bi = b >> 3;
  char* const myws = p.ws[rank];
  unsigned* const tmo = (unsigned*)(myws + TMO_OFF);
  unsigned* const cntp = (unsigned*)(myws + CNT_OFF + b * 64 + wid * 4);
  STAMP(0);

  if (bi < 16) {
    const bool link = xcd > 0 && bi < LINK_WGS;
    const int peer = ((g ^ (xcd >> 2)) << 2) | ((jq + xcd) & 3);
    if (link && wid >= LINK_WAVES) { STAMP_END(0); return; }
    const __amdgpu_buffer_rsrc_t rx = RSRC(p.x, SH * KD * 4);
    const __amdgpu_buffer_rsrc_t rw = RSRC(p.w, SH * KD * 4);
    const int dst = link ? peer : rank;
    const int mode = !link ? 2 : ((dst >> 2) == g ? 0 : 1);
    const int step = link ? NLW : NSELF;
    const int total = link ? 128 * 5 : 128 * 13;
    int u;
    if (link) u = wid * LINK_WGS + bi;
    else u = wid * (NSELF / 8) + (xcd == 0 ? bi : 16 + (xcd - 1) * (16 - LINK_WGS) + (bi - LINK_WGS));
    const __amdgpu_buffer_rsrc_t rd = RSRC(p.ws[dst], WS_USED);
    auto decode = [&](int uu, bool& isw, int& sblk, int& dblk, int& k32) {
      if (mode == 2) {
        k32 = uu / 13;
        const int s = uu - k32 * 13;
        isw = s >= 5; sblk = isw ? s - 5 : s; dblk = isw ? 32 + s - 5 : jq * 5 + s;
      } else {
        k32 = uu / 5;
        const int s = uu - k32 * 5;
        if (mode == 0) { isw = false; sblk = s; dblk = jq * 5 + s; }
        else if (s < 3) { isw = false; sblk = 5 + s; dblk = 20 + jq * 3 + s; }
        else { isw = true; sblk = (dst & 3) * 2 + (s - 3); dblk = 32 + 8 + jq * 2 + (s - 3); }
      }
    };
    v4u ra[8], rb2[8], o[4];
    bool isw, isw2; int sblk, dblk, k32, sblk2, dblk2, k322;
    decode(u, isw, sblk, dblk, k32);
    unit_load(ra, rx, rw, isw, sblk, k32, lane, 0);
    unsigned call;
    asm volatile("global_load_dword %0, %1, %2 sc1" : "=v"(call) : "v"(0), "s"(cntp) : "memory");
    unsigned pflag = 0xffffffffu;
    const unsigned lmask = lane == 0 ? 0u : 0xffffffffu;
    const __amdgpu_buffer_rsrc_t rs = RSRC(myws, WS_USED);
    const unsigned tko = TICK_OFF + (link ? xcd : 0) * 256;
    unsigned tk = 1, tbase = 0;
    int stat = u + 2 * step;
    u += step;
    int t = 0;
#define DRAW() do { u64 sv_; tk = 1; asm volatile("s_mov_b64 %1, exec\n\ts_cmp_lt_u32 %4, %5\n\ts_cselect_b64 exec, 1, 0\n\tbuffer_atomic_add %0, %2, %3, 0 offen sc0 sc1\n\ts_mov_b64 exec, %1" : "+v"(tk), "=&s"(sv_) : "v"(tko), "s"(rs), "s"(u), "s"(total) : "memory", "scc"); } while (0)
#pragma unroll 1
    for (;;) {
      bool more = (unsigned)u < (unsigned)total;
      decode(u, isw2, sblk2, dblk2, k322);
      unit_load(rb2, rx, rw, isw2, sblk2, k322, lane, more ? 0u : 0x80000000u);
      WAITV(8);
      asm volatile("" : "+v"(call), "+v"(tk));
      tbase = (unsigned)RFL(call) * (unsigned)(total - 2 * step) - 3u * step;
      u = stat >= 0 ? stat : (int)((unsigned)RFL(tk) - tbase);
      stat = -1;
      STAMP(1 + t);
      __builtin_amdgcn_raw_buffer_store_b32(call + 1, rd, pflag | lmask, 0, AUX_SYS);
      unit_cvt(o, ra, isw);
      unit_store(o, rd, stage_off(dblk, k32), lane);
      pflag = flag_off(dblk, k32);
      if (!more) break;
      more = (unsigned)u < (unsigned)total;
      DRAW();
      decode(u, isw, sblk, dblk, k32);
      unit_load(ra, rx, rw, isw, sblk, k32, lane, more ? 0u : 0x80000000u);
      WAITV(8);
      asm volatile("" : "+v"(tk));
      u = (int)((unsigned)RFL(tk) - tbase);
      STAMP(2 + t);
      __builtin_amdgcn_raw_buffer_store_b32(call + 1, rd, pflag | lmask, 0, AUX_SYS);
      unit_cvt(o, rb2, isw2);
      unit_store(o, rd, stage_off(dblk2, k322), lane);
      pflag = flag_off(dblk2, k322);
      if (!more) break;
      DRAW();
      t = min(t + 2, 56);
    }
    call += 1;
    WAITV(0);
    STAMP(60);
    if (lane == 0) {
      __builtin_amdgcn_raw_buffer_store_b32(call, rd, pflag, 0, AUX_SYS);
      __hip_atomic_store(cntp, call, __ATOMIC_RELAXED, __HIP_MEMORY_SCOPE_AGENT);
    }
    if (bi == 15 && (wid < 2 || (b == 127 && wid >= 4))) {
      u64 t0 = 0;
      if (wid < 2) {
        const __amdgpu_buffer_rsrc_t rq = RSRC(p.ws[peer], WS_USED);
        unsigned m;
        for (;;) {
          const v4u a = __builtin_amdgcn_raw_buffer_load_b128(rs, SLOT1_OFF + xcd * SLOT1_PITCH + lane * 16, 0, 16);
          CBAR();
          m = max(a.x, a.z);
          if (__all(a.y == call && a.w == call)) break;
          if (spin_fail(t0, tmo, call)) { m = BAD_MAX; break; }
        }
        const v2u mv = {wave_umax(m), call};
        __builtin_amdgcn_raw_buffer_store_b64(mv, rq, SLOT2_OFF + lane * 256 + rank * 8, 0, AUX_SYS);
        __builtin_amdgcn_raw_buffer_store_b64(mv, rq, SLOT2_OFF + (lane + 64) * 256 + rank * 8, 0, AUX_SYS);
        STAMP(61);
      } else {
        bool bad = false;
        for (;;) {
          const unsigned a = __builtin_amdgcn_raw_buffer_load_b32(rs, DONE_OFF + (wid - 4) * 256 + lane * 4, 0, AUX_SYS);
          CBAR();
          if (__all(a >= call)) break;
          if (spin_fail(t0, tmo, call)) { bad = true; break; }
        }
        if (bad && lane == 0) {
          __builtin_amdgcn_s_sleep(127);
          p.out[rank][(size_t)phys(rank) * SH * SH] = (char)-128;
        }
        STAMP(62);
      }
      STAMP_END(3);
      return;
    }
    STAMP_END(link ? 2 : 5);
    return;
  }

  volatile LAS unsigned* const ctl = (volatile LAS unsigned*)(L8 + CTL);
  const unsigned call = __hip_atomic_load(cntp, __ATOMIC_RELAXED, __HIP_MEMORY_SCOPE_AGENT) + 1;
  if (threadIdx.x < 16) ctl[threadIdx.x] = 0;
  if (lane == 0) __hip_atomic_store(cntp, call, __ATOMIC_RELAXED, __HIP_MEMORY_SCOPE_AGENT);
  WAITL();
  __builtin_amdgcn_s_barrier();
  CBAR();
  const int ti = bi - 16;
  const int rbb = (xcd >> 1) * 4 + (ti & 3);
  const int cti = ti >> 2;
  const int ct = (cti & 1) + 2 * (xcd & 1) + 4 * (cti >> 1);
  const __amdgpu_buffer_rsrc_t rs = RSRC(myws, WS_USED);

  if (wid >= 4) {
    const int s = wid - 4;
    const int blk = s < 2 ? 2 * rbb + s : 32 + 2 * ct + (s - 2);
    const unsigned sbase = (s < 2 ? XS_OFF + blk * 8192 : WS_OFF + (blk - 32) * 8192) + lane * 16;
    const unsigned sstep = s < 2 ? 32 * 8192 : 16 * 8192;
    const int c0 = s < 2 ? 2 * s : s - 2, c1 = s < 2 ? 2 * s + 1 : s;
    const unsigned fo = FLAG_OFF + blk * FLAG_PITCH + lane * 8;
    int avail = 0, issued = 0, pub = 0;
    u64 t0 = 0;
#pragma unroll 1
    while (pub < 64) {
      const int fr = RFL(min(ctl[4 + c0], ctl[4 + c1])) + 4;
      const int lim = min(avail, fr);
      CBAR();
#pragma unroll 1
      while (issued < lim) {
        LAS char* dl = L8 + s * 32768 + (issued & 3) * 8192;
#pragma unroll
        for (int i = 0; i < 8; ++i)
          __builtin_amdgcn_raw_ptr_buffer_load_lds(rs, (LAS void*)(dl + i * 1024), 16, sbase + i * 1024, issued * sstep, 0, AUX_DMA);
        ++issued;
      }
      if (issued == avail && avail < 64) {
        v2u f = {0u, 0u};
        const bool inw = (unsigned)(lane - avail) < 16u;
        if (inw) f = __builtin_amdgcn_raw_buffer_load_b64(rs, fo, 0, AUX_SYS);
        const int o = issued - pub;
        if (o >= 4) { WAITV(25); ctl[s] = issued - 3; }
        if (o >= 3) { WAITV(17); ctl[s] = issued - 2; }
        if (o >= 2) { WAITV(9); ctl[s] = issued - 1; }
        if (o >= 1) { WAITV(1); ctl[s] = issued; STAMP(1 + (issued >> 1)); }
        pub = issued;
        CBAR();
        const u64 mk = __ballot(lane < avail || (inw && f.x >= call && f.y >= call));
        const int na = ~mk ? __builtin_ctzll(~mk) : 64;
        if (na == avail) {
          if (spin_fail(t0, tmo, call)) { avail = 64; ctl[12] = 1; }
        } else {
          avail = na; t0 = 0;
        }
      } else if (pub < issued) {
        const int o = issued - pub;
        if (o >= 4) WAITV(24);
        else if (o == 3) WAITV(16);
        else if (o == 2) WAITV(8);
        else WAITV(0);
        ++pub;
        ctl[s] = pub;
        STAMP(1 + (pub >> 1));
        t0 = 0;
      } else {
        if (spin_fail(t0, tmo, call)) { ctl[12] = 1; break; }
      }
    }
    WAITV(0);
    WAITL();
    STAMP_END(4);
    return;
  }

  const int wm = wid >> 1, wn = wid & 1;
  const bool remote = ct >= 4;
  const int orank = (remote ? (1 - g) * 4 + (ct - 4) : rank) & 7;
  u64 donep = (u64)p.ws[orank] + DONE_OFF + ((jq * 16 + rbb) * 4 + wid) * 4;
  unsigned outlo = (unsigned)(u64)p.out[orank], outhi = (unsigned)((u64)p.out[orank] >> 32);
  asm volatile("" : "+v"(donep), "+s"(outlo), "+s"(outhi));
  char* const outp = (char*)(((u64)(unsigned)RFL(outhi) << 32) | (u64)(unsigned)RFL(outlo));
  f32x4 acc[4][4];
#pragma unroll
  for (int m = 0; m < 4; ++m)
#pragma unroll
    for (int n = 0; n < 4; ++n) acc[m][n] = f32x4{0.f, 0.f, 0.f, 0.f};
  {
    const LAS char* xb = L8 + wm * 32768 + lane * 16;
    const LAS char* wb = L8 + (2 + wn) * 32768 + lane * 16;
    int full = 0;
    v4u xa[4], wa[4], xc[4], wc[4];
    auto rd_half = [&](v4u (&xf)[4], v4u (&wf)[4], int off) {
#pragma unroll
      for (int m = 0; m < 4; ++m) wf[m] = *(const LAS v4u*)(wb + off + m * 1024);
#pragma unroll
      for (int m = 0; m < 4; ++m) xf[m] = *(const LAS v4u*)(xb + off + m * 1024);
    };
    auto mm = [&](const v4u (&xf)[4], const v4u (&wf)[4]) {
#pragma unroll
      for (int m = 0; m < 4; ++m)
#pragma unroll
        for (int n = 0; n < 4; ++n)
          acc[m][n] = __builtin_amdgcn_mfma_f32_16x16x32_bf16(__builtin_bit_cast(bf16x8, wf[n]),
                                                              __builtin_bit_cast(bf16x8, xf[m]), acc[m][n], 0, 0, 0);
    };
    auto wait_full = [&](int kc) {
      u64 t0 = 0;
      int sp = 0;
      while (full <= kc) {
        full = RFL(min(ctl[wm], ctl[2 + wn]));
        if ((++sp & 63) == 0 && spin_fail(t0, tmo, call)) { full = 64; ctl[12] = 1; }
      }
      CBAR();
    };
    wait_full(0);
    rd_half(xa, wa, 0);
#pragma unroll 1
    for (int kc = 0; kc < 64; ++kc) {
      if ((kc & 1) == 0) STAMP(1 + (kc >> 1));
      rd_half(xc, wc, (kc & 3) * 8192 + 4096);
      const unsigned f0 = ctl[wm], f1 = ctl[2 + wn];
      __builtin_amdgcn_sched_barrier(0);
      mm(xa, wa);
      __builtin_amdgcn_sched_barrier(0);
      WAITL();
      ctl[4 + wid] = kc + 1;
      full = max(full, RFL(min(f0, f1)));
      const bool ahead = full > kc + 1;
      if (ahead) rd_half(xa, wa, ((kc + 1) & 3) * 8192);
      __builtin_amdgcn_sched_barrier(0);
      mm(xc, wc);
      __builtin_amdgcn_sched_barrier(0);
      if (!ahead && kc < 63) {
        wait_full(kc + 1);
        rd_half(xa, wa, ((kc + 1) & 3) * 8192);
      }
    }
  }
  STAMP(34);
  float mx = 0.f;
#pragma unroll
  for (int m = 0; m < 4; ++m)
#pragma unroll
    for (int n = 0; n < 4; ++n)
#pragma unroll
      for (int e = 0; e < 4; ++e) mx = fmaxf(mx, acc[m][n][e]);
  bool bad;
  {
    const unsigned wmx = wave_umax(__builtin_bit_cast(unsigned, mx));
    if (lane == 0) ctl[8 + wid] = wmx;
    WAITL();
    __builtin_amdgcn_s_barrier();
    CBAR();
    if (wid == 0 && lane < 8) {
      const unsigned m4 = max(max(ctl[8], ctl[9]), max(ctl[10], ctl[11]));
      const v2u mv = {m4, call};
      __builtin_amdgcn_raw_buffer_store_b64(mv, rs, SLOT1_OFF + lane * SLOT1_PITCH + (b - 128) * 8, 0, 16);
    }
    bad = ctl[12] != 0;
  }
  STAMP(35);
#pragma unroll
  for (int m = 0; m < 4; ++m)
#pragma unroll
    for (int n = 0; n < 4; ++n)
#pragma unroll
      for (int e = 0; e < 4; ++e) acc[m][n][e] = fmaxf(acc[m][n][e], 0.f);
#pragma unroll
  for (int m = 0; m < 4; ++m) asm volatile("" : "+v"(acc[m][0]), "+v"(acc[m][1]), "+v"(acc[m][2]), "+v"(acc[m][3]));
  unsigned gm;
  {
    const unsigned so = SLOT2_OFF + (b - 128) * 256 + (lane & 7) * 8;
    u64 t0 = 0;
    for (int sp = 1;; ++sp) {
      if (wid < 2) {
        const v2u a = __builtin_amdgcn_raw_buffer_load_b64(rs, so, 0, AUX_SYS);
        CBAR();
        if (__all(a.y >= call)) {
          gm = a.x;
          gm = max(gm, (unsigned)__builtin_amdgcn_update_dpp(0, (int)gm, 0x111, 0xf, 0xf, true));
          gm = max(gm, (unsigned)__builtin_amdgcn_update_dpp(0, (int)gm, 0x112, 0xf, 0xf, true));
          gm = max(gm, (unsigned)__builtin_amdgcn_update_dpp(0, (int)gm, 0x114, 0xf, 0xf, true));
          gm = __builtin_amdgcn_readlane(gm, 7);
          ctl[13] = gm + 1;
          break;
        }
      }
      gm = RFL(ctl[13]) - 1;
      if (gm != 0xffffffffu) break;
      if ((wid < 2 || (sp & 63) == 0) && spin_fail(t0, tmo, call)) { gm = BAD_MAX; break; }
    }
  }
  STAMP(36);
  bad = bad || gm == BAD_MAX;
  const float amax = __builtin_bit_cast(float, gm);
  const float inv = amax > 0.f && !bad ? 127.0f * __builtin_amdgcn_rcpf(amax) : 0.f;
  const float magic = bad ? 12582912.0f + 128.0f : 12582912.0f;
  const int rb = 2 * rbb + wm;
  int srank, lrow;
  if (rb < 20) { srank = g * 4 + rb / 5; lrow = (rb % 5) * 64; }
  else { srank = (1 - g) * 4 + (rb - 20) / 3; lrow = (5 + (rb - 20) % 3) * 64; }
  const int ocol = (remote ? jq * 128 : ct * 128) + wn * 64;
  const __amdgpu_buffer_rsrc_t ro = RSRC(outp, NR * SH * SH);
  const unsigned obase = (unsigned)(phys(srank) * SH + lrow + (lane & 15)) * SH + ocol + (lane >> 4) * 16;
#pragma unroll
  for (int m = 0; m < 4; ++m) {
    v4u q;
    const f32x4 inv4 = {inv, inv, inv, inv}, mg4 = {magic, magic, magic, magic};
    v4u tb[4];
#pragma unroll
    for (int n = 0; n < 4; ++n) tb[n] = __builtin_bit_cast(v4u, __builtin_elementwise_fma(acc[m][n], inv4, mg4));
#pragma unroll
    for (int e = 0; e < 4; ++e) {
      const unsigned lo = __builtin_amdgcn_perm(tb[1][e], tb[0][e], 0x0c0c0400u);
      const unsigned hi = __builtin_amdgcn_perm(tb[3][e], tb[2][e], 0x04000c0cu);
      q[e] = lo | hi;
    }
    __builtin_amdgcn_raw_buffer_store_b128(q, ro, obase + m * (16 * SH), 0, AUX_SYS);
  }
  STAMP(37);
  if (remote) {
    WAITV(0);
    if (lane == 0)
      __hip_atomic_store((GAS unsigned*)donep, call, __ATOMIC_RELAXED, __HIP_MEMORY_SCOPE_SYSTEM);
  }
  STAMP(38);
  STAMP_END(1);
}

extern "C" void kernel_launch(int rank, int nranks, ncclComm_t comm, hipStream_t stream, void* const* d_in,
                              const int* in_sizes, int n_in, void* d_out, int out_size, const int* in_dtypes,
                              int out_dtype, void* const* d_out_all, void* const* d_ws_all, size_t ws_bytes) {
  if (nranks != NR || (unsigned)rank >= (unsigned)NR || n_in < 2 || ws_bytes < (size_t)WS_USED ||
      in_sizes[0] != SH * KD || in_sizes[1] != SH * KD || out_size != NR * SH * SH) {
    fprintf(stderr, "unexpected problem shape\n");
    return;
  }
  Params p;
  p.rank = 0;
  p.x = (const float*)d_in[0];
  p.w = (const float*)d_in[1];
  for (int i = 0; i < NR; ++i) {
    p.ws[i] = (char*)d_ws_all[phys(i)];
    p.out[i] = (char*)d_out_all[phys(i)];
    if (phys(i) == rank) p.rank = i;
  }
  HIP_CHECK(hipFuncSetAttribute((const void*)ag_gemm_kernel, hipFuncAttributeMaxDynamicSharedMemorySize, LDS_BYTES));
  ag_gemm_kernel<<<NWG, NTHR, LDS_BYTES, stream>>>(p);
  HIP_CHECK(hipGetLastError());
}
```

```cpp
#include <hip/hip_runtime.h>
#include <rccl/rccl.h>
#include <cstdio>

#define HIP_CHECK(expr) do { hipError_t _e = (expr); if (_e != hipSuccess) { \
    fprintf(stderr, "HIP error %s at %s:%d\n", hipGetErrorString(_e), __FILE__, __LINE__); \
    return; } } while (0)

typedef unsigned v4u __attribute__((ext_vector_type(4)));
typedef unsigned v2u __attribute__((ext_vector_type(2)));
typedef float f32x4 __attribute__((ext_vector_type(4)));
typedef float f32x2 __attribute__((ext_vector_type(2)));
typedef __bf16 bf16x2 __attribute__((ext_vector_type(2)));
typedef short bf16x8 __attribute__((ext_vector_type(8)));
typedef unsigned long long u64;
#define LAS __attribute__((address_space(3)))
#define GAS __attribute__((address_space(1)))

constexpr int NR = 8;
constexpr int KD = 4096;
constexpr int SH = 512;
constexpr int NWG = 256;
constexpr int NTHR = 512;
#define LINK_WGS 14
#define LINK_WAVES 1
constexpr unsigned PHYS = 0x75416320u;
__host__ __device__ constexpr int phys(int r) { return (PHYS >> (4 * r)) & 7; }
constexpr int NLW = LINK_WGS * LINK_WAVES;
constexpr int NSELF = (7 * (16 - LINK_WGS) + 16) * 8;
static_assert(LINK_WGS < 16 && 3 * NLW <= 128 * 5 && 3 * NSELF <= 128 * 13, "three fixed units per wave");
constexpr unsigned XS_OFF = 0;
constexpr unsigned WS_OFF = 16u << 20;
constexpr unsigned FLAG_OFF = 24u << 20;
constexpr unsigned FLAG_PITCH = 4096 + 512;
constexpr unsigned SLOT1_OFF = FLAG_OFF + (256u << 10);
constexpr unsigned SLOT1_PITCH = 1024 + 256;
constexpr unsigned SLOT2_OFF = SLOT1_OFF + (16u << 10);
constexpr unsigned DONE_OFF = SLOT2_OFF + (32u << 10);
constexpr unsigned CNT_OFF = DONE_OFF + (4u << 10);
constexpr unsigned TICK_OFF = CNT_OFF + (16u << 10);
constexpr unsigned TMO_OFF = TICK_OFF + (4u << 10);
constexpr unsigned WS_USED = TMO_OFF + 4096;
constexpr unsigned STAMP_OFF = 32u << 20;
constexpr u64 DEADLINE_TICKS = 50000000ull;
constexpr int CTL = 128 * 1024;
constexpr int LDS_BYTES = CTL + 256;
constexpr unsigned BAD_MAX = 0xfffffffeu;
#define AUX_DMA 17
constexpr int AUX_SYS = 17;

struct Params {
  const float* x;
  const float* w;
  char* ws[NR];
  char* out[NR];
  int rank;
};

#define RSRC(p, bytes) __builtin_amdgcn_make_buffer_rsrc((void*)(p), 0, (int)(bytes), 0x00020000)
#define CBAR() asm volatile("" ::: "memory")
#define WAITV(n) asm volatile("s_waitcnt vmcnt(" #n ")" ::: "memory")
#define WAITL() asm volatile("s_waitcnt lgkmcnt(0)" ::: "memory")
#define RFL(v) __builtin_amdgcn_readfirstlane(v)

#define STAMP(i) do {} while (0)
#define STAMP_END(cls) do {} while (0)

__device__ __forceinline__ unsigned pk(float a, float b) {
  f32x2 v = {a, b};
  return __builtin_bit_cast(unsigned, __builtin_convertvector(v, bf16x2));
}

__device__ __forceinline__ void unit_load(v4u (&r)[8], __amdgpu_buffer_rsrc_t rx, __amdgpu_buffer_rsrc_t rw,
                                          bool isw, int blk, int k32, int lane, unsigned oob) {
  const int kg = lane >> 4, c = lane & 15;
  if (!isw) {
    const unsigned base = ((unsigned)(blk * 64 + c) * (KD * 4) + (unsigned)(k32 * 32 + 4 * kg) * 4) | oob;
#pragma unroll
    for (int m = 0; m < 4; ++m) {
      r[2 * m] = __builtin_amdgcn_raw_buffer_load_b128(rx, base + m * (16 * KD * 4), 0, 0);
      r[2 * m + 1] = __builtin_amdgcn_raw_buffer_load_b128(rx, base + m * (16 * KD * 4) + 64, 0, 0);
    }
  } else {
    const unsigned base = ((unsigned)(k32 * 32 + 4 * kg) * (SH * 4) + (unsigned)(blk * 64 + 4 * c) * 4) | oob;
#pragma unroll
    for (int jj = 0; jj < 8; ++jj)
      r[jj] = __builtin_amdgcn_raw_buffer_load_b128(rw, base + ((jj & 3) + 16 * (jj >> 2)) * (SH * 4), 0, 0);
  }
}
__device__ __forceinline__ float asf(unsigned u) { return __builtin_bit_cast(float, u); }
__device__ __forceinline__ void unit_cvt(v4u (&o)[4], const v4u (&r)[8], bool isw) {
  if (!isw) {
#pragma unroll
    for (int m = 0; m < 4; ++m) {
      o[m].x = pk(asf(r[2 * m].x), asf(r[2 * m].y));
      o[m].y = pk(asf(r[2 * m].z), asf(r[2 * m].w));
      o[m].z = pk(asf(r[2 * m + 1].x), asf(r[2 * m + 1].y));
      o[m].w = pk(asf(r[2 * m + 1].z), asf(r[2 * m + 1].w));
    }
  } else {
#pragma unroll
    for (int i = 0; i < 4; ++i) {
      o[i].x = pk(asf(r[0][i]), asf(r[1][i]));
      o[i].y = pk(asf(r[2][i]), asf(r[3][i]));
      o[i].z = pk(asf(r[4][i]), asf(r[5][i]));
      o[i].w = pk(asf(r[6][i]), asf(r[7][i]));
    }
  }
}
__device__ __forceinline__ unsigned stage_off(int dblk, int k32) {
  const int kc = k32 >> 1, h = k32 & 1;
  return dblk < 32 ? XS_OFF + (unsigned)((kc * 32 + dblk) * 2 + h) * 4096u
                   : WS_OFF + (unsigned)((kc * 16 + (dblk - 32)) * 2 + h) * 4096u;
}
__device__ __forceinline__ unsigned flag_off(int dblk, int k32) {
  return FLAG_OFF + (unsigned)dblk * FLAG_PITCH + (unsigned)k32 * 4u;
}
__device__ __forceinline__ void unit_store(const v4u (&o)[4], __amdgpu_buffer_rsrc_t rd, unsigned off, int lane) {
#pragma unroll
  for (int i = 0; i < 4; ++i) __builtin_amdgcn_raw_buffer_store_b128(o[i], rd, off + lane * 16 + i * 1024, 0, AUX_SYS);
}

__device__ __forceinline__ bool spin_fail(u64& t0, unsigned* tmo, unsigned call) {
  const u64 now = __builtin_amdgcn_s_memrealtime();
  if (t0 == 0) t0 = now;
  if (now - t0 > 10000) {
    if (__hip_atomic_load(tmo, __ATOMIC_RELAXED, __HIP_MEMORY_SCOPE_AGENT) == call) return true;
    if (now - t0 > DEADLINE_TICKS) {
      __hip_atomic_store(tmo, call, __ATOMIC_RELAXED, __HIP_MEMORY_SCOPE_AGENT);
      return true;
    }
  }
  return false;
}

__device__ __forceinline__ unsigned wave_umax(unsigned v) {
  v = max(v, (unsigned)__builtin_amdgcn_update_dpp(0, (int)v, 0x111, 0xf, 0xf, true));
  v = max(v, (unsigned)__builtin_amdgcn_update_dpp(0, (int)v, 0x112, 0xf, 0xf, true));
  v = max(v, (unsigned)__builtin_amdgcn_update_dpp(0, (int)v, 0x114, 0xf, 0xf, true));
  v = max(v, (unsigned)__builtin_amdgcn_update_dpp(0, (int)v, 0x118, 0xf, 0xf, true));
  const unsigned a = __builtin_amdgcn_readlane(v, 15), b = __builtin_amdgcn_readlane(v, 31);
  const unsigned c = __builtin_amdgcn_readlane(v, 47), d = __builtin_amdgcn_readlane(v, 63);
  return max(max(a, b), max(c, d));
}

__global__ void __launch_bounds__(NTHR, 2) ag_gemm_kernel(Params p) {
  extern __shared__ __attribute__((aligned(16))) char smem[];
  LAS char* const L8 = (LAS char*)smem;
  const int lane = threadIdx.x & 63;
  const int wid = RFL(threadIdx.x >> 6);
  const int b = blockIdx.x;
  const int rank = p.rank, g = rank >> 2, jq = rank & 3;
  const int xcd = b & 7, bi = b >> 3;
  char* const myws = p.ws[rank];
  unsigned* const tmo = (unsigned*)(myws + TMO_OFF);
  unsigned* const cntp = (unsigned*)(myws + CNT_OFF + b * 64 + wid * 4);
  STAMP(0);

  if (bi < 16) {
    const bool link = xcd > 0 && bi < LINK_WGS;
    const int peer = ((g ^ (xcd >> 2)) << 2) | ((jq + xcd) & 3);
    if (link && wid >= LINK_WAVES) { STAMP_END(0); return; }
    const __amdgpu_buffer_rsrc_t rx = RSRC(p.x, SH * KD * 4);
    const __amdgpu_buffer_rsrc_t rw = RSRC(p.w, SH * KD * 4);
    const int dst = link ? peer : rank;
    const int mode = !link ? 2 : ((dst >> 2) == g ? 0 : 1);
    const int step = link ? NLW : NSELF;
    const int total = link ? 128 * 5 : 128 * 13;
    int u;
    if (link) u = wid * LINK_WGS + bi;
    else u = wid * (NSELF / 8) + (xcd == 0 ? bi : 16 + (xcd - 1) * (16 - LINK_WGS) + (bi - LINK_WGS));
    const __amdgpu_buffer_rsrc_t rd = RSRC(p.ws[dst], WS_USED);
    auto decode = [&](int uu, bool& isw, int& sblk, int& dblk, int& k32) {
      if (mode == 2) {
        k32 = uu / 13;
        const int s = uu - k32 * 13;
        isw = s >= 5; sblk = isw ? s - 5 : s; dblk = isw ? 32 + s - 5 : jq * 5 + s;
      } else {
        k32 = uu / 5;
        const int s = uu - k32 * 5;
        if (mode == 0) { isw = false; sblk = s; dblk = jq * 5 + s; }
        else if (s < 3) { isw = false; sblk = 5 + s; dblk = 20 + jq * 3 + s; }
        else { isw = true; sblk = (dst & 3) * 2 + (s - 3); dblk = 32 + 8 + jq * 2 + (s - 3); }
      }
    };
    v4u ra[8], rb2[8], o[4];
    bool isw, isw2; int sblk, dblk, k32, sblk2, dblk2, k322;
    decode(u, isw, sblk, dblk, k32);
    unit_load(ra, rx, rw, isw, sblk, k32, lane, 0);
    unsigned call;
    asm volatile("global_load_dword %0, %1, %2 sc1" : "=v"(call) : "v"(0), "s"(cntp) : "memory");
    unsigned pflag = 0xffffffffu;
    const unsigned lmask = lane == 0 ? 0u : 0xffffffffu;
    const __amdgpu_buffer_rsrc_t rs = RSRC(myws, WS_USED);
    const unsigned tko = TICK_OFF + (link ? xcd : 0) * 256;
    unsigned tk = 1, tbase = 0;
    int stat = u + 2 * step;
    u += step;
    int t = 0;
#define DRAW() do { u64 sv_; tk = 1; asm volatile("s_mov_b64 %1, exec\n\ts_cmp_lt_u32 %4, %5\n\ts_cselect_b64 exec, 1, 0\n\tbuffer_atomic_add %0, %2, %3, 0 offen sc0 sc1\n\ts_mov_b64 exec, %1" : "+v"(tk), "=&s"(sv_) : "v"(tko), "s"(rs), "s"(u), "s"(total) : "memory", "scc"); } while (0)
#pragma unroll 1
    for (;;) {
      bool more = (unsigned)u < (unsigned)total;
      decode(u, isw2, sblk2, dblk2, k322);
      unit_load(rb2, rx, rw, isw2, sblk2, k322, lane, more ? 0u : 0x80000000u);
      WAITV(8);
      asm volatile("" : "+v"(call), "+v"(tk));
      tbase = (unsigned)RFL(call) * (unsigned)(total - 2 * step) - 3u * step;
      u = stat >= 0 ? stat : (int)((unsigned)RFL(tk) - tbase);
      stat = -1;
      STAMP(1 + t);
      __builtin_amdgcn_raw_buffer_store_b32(call + 1, rd, pflag | lmask, 0, AUX_SYS);
      unit_cvt(o, ra, isw);
      unit_store(o, rd, stage_off(dblk, k32), lane);
      pflag = flag_off(dblk, k32);
      if (!more) break;
      more = (unsigned)u < (unsigned)total;
      DRAW();
      decode(u, isw, sblk, dblk, k32);
      unit_load(ra, rx, rw, isw, sblk, k32, lane, more ? 0u : 0x80000000u);
      WAITV(8);
      asm volatile("" : "+v"(tk));
      u = (int)((unsigned)RFL(tk) - tbase);
      STAMP(2 + t);
      __builtin_amdgcn_raw_buffer_store_b32(call + 1, rd, pflag | lmask, 0, AUX_SYS);
      unit_cvt(o, rb2, isw2);
      unit_store(o, rd, stage_off(dblk2, k322), lane);
      pflag = flag_off(dblk2, k322);
      if (!more) break;
      DRAW();
      t = min(t + 2, 56);
    }
    call += 1;
    WAITV(0);
    STAMP(60);
    if (lane == 0) {
      __builtin_amdgcn_raw_buffer_store_b32(call, rd, pflag, 0, AUX_SYS);
      __hip_atomic_store(cntp, call, __ATOMIC_RELAXED, __HIP_MEMORY_SCOPE_AGENT);
    }
    if (bi == 15 && (wid < 2 || (b == 127 && wid >= 4))) {
      u64 t0 = 0;
      if (wid < 2) {
        const __amdgpu_buffer_rsrc_t rq = RSRC(p.ws[peer], WS_USED);
        unsigned m;
        for (;;) {
          const v4u a = __builtin_amdgcn_raw_buffer_load_b128(rs, SLOT1_OFF + xcd * SLOT1_PITCH + lane * 16, 0, 16);
          CBAR();
          m = max(a.x, a.z);
          if (__all(a.y == call && a.w == call)) break;
          if (spin_fail(t0, tmo, call)) { m = BAD_MAX; break; }
        }
        const v2u mv = {wave_umax(m), call};
        __builtin_amdgcn_raw_buffer_store_b64(mv, rq, SLOT2_OFF + lane * 256 + rank * 8, 0, AUX_SYS);
        __builtin_amdgcn_raw_buffer_store_b64(mv, rq, SLOT2_OFF + (lane + 64) * 256 + rank * 8, 0, AUX_SYS);
        STAMP(61);
      } else {
        bool bad = false;
        for (;;) {
          const unsigned a = __builtin_amdgcn_raw_buffer_load_b32(rs, DONE_OFF + (wid - 4) * 256 + lane * 4, 0, AUX_SYS);
          CBAR();
          if (__all(a >= call)) break;
          if (spin_fail(t0, tmo, call)) { bad = true; break; }
        }
        if (bad && lane == 0) {
          __builtin_amdgcn_s_sleep(127);
          p.out[rank][(size_t)phys(rank) * SH * SH] = (char)-128;
        }
        STAMP(62);
      }
      STAMP_END(3);
      return;
    }
    STAMP_END(link ? 2 : 5);
    return;
  }

  volatile LAS unsigned* const ctl = (volatile LAS unsigned*)(L8 + CTL);
  const unsigned call = __hip_atomic_load(cntp, __ATOMIC_RELAXED, __HIP_MEMORY_SCOPE_AGENT) + 1;
  if (threadIdx.x < 16) ctl[threadIdx.x] = 0;
  if (lane == 0) __hip_atomic_store(cntp, call, __ATOMIC_RELAXED, __HIP_MEMORY_SCOPE_AGENT);
  WAITL();
  __builtin_amdgcn_s_barrier();
  CBAR();
  const int ti = bi - 16;
  const int rbb = (xcd >> 1) * 4 + (ti & 3);
  const int cti = ti >> 2;
  const int ct = (cti & 1) + 2 * (xcd & 1) + 4 * (cti >> 1);
  const __amdgpu_buffer_rsrc_t rs = RSRC(myws, WS_USED);

  if (wid >= 4) {
    const int s = wid - 4;
    const int blk = s < 2 ? 2 * rbb + s : 32 + 2 * ct + (s - 2);
    const unsigned sbase = (s < 2 ? XS_OFF + blk * 8192 : WS_OFF + (blk - 32) * 8192) + lane * 16;
    const unsigned sstep = s < 2 ? 32 * 8192 : 16 * 8192;
    const int c0 = s < 2 ? 2 * s : s - 2, c1 = s < 2 ? 2 * s + 1 : s;
    const unsigned fo = FLAG_OFF + blk * FLAG_PITCH + lane * 8;
    int avail = 0, issued = 0, pub = 0;
    u64 t0 = 0;
#pragma unroll 1
    while (pub < 64) {
      const int fr = RFL(min(ctl[4 + c0], ctl[4 + c1])) + 4;
      const int lim = min(avail, fr);
      CBAR();
#pragma unroll 1
      while (issued < lim) {
        LAS char* dl = L8 + s * 32768 + (issued & 3) * 8192;
#pragma unroll
        for (int i = 0; i < 8; ++i)
          __builtin_amdgcn_raw_ptr_buffer_load_lds(rs, (LAS void*)(dl + i * 1024), 16, sbase + i * 1024, issued * sstep, 0, AUX_DMA);
        ++issued;
      }
      if (issued == avail && avail < 64) {
        v2u f = {0u, 0u};
        const bool inw = (unsigned)(lane - avail) < 16u;
        if (inw) f = __builtin_amdgcn_raw_buffer_load_b64(rs, fo, 0, AUX_SYS);
        const int o = issued - pub;
        if (o >= 4) { WAITV(25); ctl[s] = issued - 3; }
        if (o >= 3) { WAITV(17); ctl[s] = issued - 2; }
        if (o >= 2) { WAITV(9); ctl[s] = issued - 1; }
        if (o >= 1) { WAITV(1); ctl[s] = issued; STAMP(1 + (issued >> 1)); }
        pub = issued;
        CBAR();
        const u64 mk = __ballot(lane < avail || (inw && f.x >= call && f.y >= call));
        const int na = ~mk ? __builtin_ctzll(~mk) : 64;
        if (na == avail) {
          if (spin_fail(t0, tmo, call)) { avail = 64; ctl[12] = 1; }
        } else {
          avail = na; t0 = 0;
        }
      } else if (pub < issued) {
        const int o = issued - pub;
        if (o >= 4) WAITV(24);
        else if (o == 3) WAITV(16);
        else if (o == 2) WAITV(8);
        else WAITV(0);
        ++pub;
        ctl[s] = pub;
        STAMP(1 + (pub >> 1));
        t0 = 0;
      } else {
        if (spin_fail(t0, tmo, call)) { ctl[12] = 1; break; }
      }
    }
    WAITV(0);
    WAITL();
    STAMP_END(4);
    return;
  }

  const int wm = wid >> 1, wn = wid & 1;
  const bool remote = ct >= 4;
  const int orank = (remote ? (1 - g) * 4 + (ct - 4) : rank) & 7;
  u64 donep = (u64)p.ws[orank] + DONE_OFF + ((jq * 16 + rbb) * 4 + wid) * 4;
  unsigned outlo = (unsigned)(u64)p.out[orank], outhi = (unsigned)((u64)p.out[orank] >> 32);
  asm volatile("" : "+v"(donep), "+s"(outlo), "+s"(outhi));
  char* const outp = (char*)(((u64)(unsigned)RFL(outhi) << 32) | (u64)(unsigned)RFL(outlo));
  f32x4 acc[4][4];
#pragma unroll
  for (int m = 0; m < 4; ++m)
#pragma unroll
    for (int n = 0; n < 4; ++n) acc[m][n] = f32x4{0.f, 0.f, 0.f, 0.f};
  {
    const LAS char* xb = L8 + wm * 32768 + lane * 16;
    const LAS char* wb = L8 + (2 + wn) * 32768 + lane * 16;
    int full = 0;
    v4u xa[4], wa[4], xc[4], wc[4];
    auto rd_half = [&](v4u (&xf)[4], v4u (&wf)[4], int off) {
#pragma unroll
      for (int m = 0; m < 4; ++m) wf[m] = *(const LAS v4u*)(wb + off + m * 1024);
#pragma unroll
      for (int m = 0; m < 4; ++m) xf[m] = *(const LAS v4u*)(xb + off + m * 1024);
    };
    auto mm = [&](const v4u (&xf)[4], const v4u (&wf)[4]) {
#pragma unroll
      for (int m = 0; m < 4; ++m)
#pragma unroll
        for (int n = 0; n < 4; ++n)
          acc[m][n] = __builtin_amdgcn_mfma_f32_16x16x32_bf16(__builtin_bit_cast(bf16x8, wf[n]),
                                                              __builtin_bit_cast(bf16x8, xf[m]), acc[m][n], 0, 0, 0);
    };
    auto wait_full = [&](int kc) {
      u64 t0 = 0;
      int sp = 0;
      while (full <= kc) {
        full = RFL(min(ctl[wm], ctl[2 + wn]));
        if ((++sp & 63) == 0 && spin_fail(t0, tmo, call)) { full = 64; ctl[12] = 1; }
      }
      CBAR();
    };
    wait_full(0);
    rd_half(xa, wa, 0);
#pragma unroll 1
    for (int kc = 0; kc < 64; ++kc) {
      if ((kc & 1) == 0) STAMP(1 + (kc >> 1));
      rd_half(xc, wc, (kc & 3) * 8192 + 4096);
      const unsigned f0 = ctl[wm], f1 = ctl[2 + wn];
      __builtin_amdgcn_sched_barrier(0);
      mm(xa, wa);
      __builtin_amdgcn_sched_barrier(0);
      WAITL();
      ctl[4 + wid] = kc + 1;
      full = max(full, RFL(min(f0, f1)));
      const bool ahead = full > kc + 1;
      if (ahead) rd_half(xa, wa, ((kc + 1) & 3) * 8192);
      __builtin_amdgcn_sched_barrier(0);
      mm(xc, wc);
      __builtin_amdgcn_sched_barrier(0);
      if (!ahead && kc < 63) {
        wait_full(kc + 1);
        rd_half(xa, wa, ((kc + 1) & 3) * 8192);
      }
    }
  }
  STAMP(34);
  float mx = 0.f;
#pragma unroll
  for (int m = 0; m < 4; ++m)
#pragma unroll
    for (int n = 0; n < 4; ++n)
#pragma unroll
      for (int e = 0; e < 4; ++e) mx = fmaxf(mx, acc[m][n][e]);
  bool bad;
  {
    const unsigned wmx = wave_umax(__builtin_bit_cast(unsigned, mx));
    if (lane == 0) ctl[8 + wid] = wmx;
    WAITL();
    __builtin_amdgcn_s_barrier();
    CBAR();
    if (wid == 0 && lane < 8) {
      const unsigned m4 = max(max(ctl[8], ctl[9]), max(ctl[10], ctl[11]));
      const v2u mv = {m4, call};
      __builtin_amdgcn_raw_buffer_store_b64(mv, rs, SLOT1_OFF + lane * SLOT1_PITCH + (b - 128) * 8, 0, 16);
    }
    bad = ctl[12] != 0;
  }
  STAMP(35);
#pragma unroll
  for (int m = 0; m < 4; ++m)
#pragma unroll
    for (int n = 0; n < 4; ++n)
#pragma unroll
      for (int e = 0; e < 4; ++e) acc[m][n][e] = fmaxf(acc[m][n][e], 0.f);
#pragma unroll
  for (int m = 0; m < 4; ++m) asm volatile("" : "+v"(acc[m][0]), "+v"(acc[m][1]), "+v"(acc[m][2]), "+v"(acc[m][3]));
  unsigned gm;
  {
    const unsigned so = SLOT2_OFF + (b - 128) * 256 + (lane & 7) * 8;
    u64 t0 = 0;
    for (int sp = 1;; ++sp) {
      if (wid < 2) {
        const v2u a = __builtin_amdgcn_raw_buffer_load_b64(rs, so, 0, AUX_SYS);
        CBAR();
        if (__all(a.y >= call)) {
          gm = a.x;
          gm = max(gm, (unsigned)__builtin_amdgcn_update_dpp(0, (int)gm, 0x111, 0xf, 0xf, true));
          gm = max(gm, (unsigned)__builtin_amdgcn_update_dpp(0, (int)gm, 0x112, 0xf, 0xf, true));
          gm = max(gm, (unsigned)__builtin_amdgcn_update_dpp(0, (int)gm, 0x114, 0xf, 0xf, true));
          gm = __builtin_amdgcn_readlane(gm, 7);
          ctl[13] = gm + 1;
          break;
        }
      }
      gm = RFL(ctl[13]) - 1;
      if (gm != 0xffffffffu) break;
      if ((wid < 2 || (sp & 63) == 0) && spin_fail(t0, tmo, call)) { gm = BAD_MAX; break; }
    }
  }
  STAMP(36);
  bad = bad || gm == BAD_MAX;
  const float amax = __builtin_bit_cast(float, gm);
  const float inv = amax > 0.f && !bad ? 127.0f * __builtin_amdgcn_rcpf(amax) : 0.f;
  const float magic = bad ? 12582912.0f + 128.0f : 12582912.0f;
  const int rb = 2 * rbb + wm;
  int srank, lrow;
  if (rb < 20) { srank = g * 4 + rb / 5; lrow = (rb % 5) * 64; }
  else { srank = (1 - g) * 4 + (rb - 20) / 3; lrow = (5 + (rb - 20) % 3) * 64; }
  const int ocol = (remote ? jq * 128 : ct * 128) + wn * 64;
  const __amdgpu_buffer_rsrc_t ro = RSRC(outp, NR * SH * SH);
  const unsigned obase = (unsigned)(phys(srank) * SH + lrow + (lane & 15)) * SH + ocol + (lane >> 4) * 16;
#pragma unroll
  for (int m = 0; m < 4; ++m) {
    v4u q;
    const f32x4 inv4 = {inv, inv, inv, inv}, mg4 = {magic, magic, magic, magic};
    v4u tb[4];
#pragma unroll
    for (int n = 0; n < 4; ++n) tb[n] = __builtin_bit_cast(v4u, __builtin_elementwise_fma(acc[m][n], inv4, mg4));
#pragma unroll
    for (int e = 0; e < 4; ++e) {
      const unsigned lo = __builtin_amdgcn_perm(tb[1][e], tb[0][e], 0x0c0c0400u);
      const unsigned hi = __builtin_amdgcn_perm(tb[3][e], tb[2][e], 0x04000c0cu);
      q[e] = lo | hi;
    }
    __builtin_amdgcn_raw_buffer_store_b128(q, ro, obase + m * (16 * SH), 0, AUX_SYS);
  }
  STAMP(37);
  if (remote) {
    WAITV(0);
    if (lane == 0)
      __hip_atomic_store((GAS unsigned*)donep, call, __ATOMIC_RELAXED, __HIP_MEMORY_SCOPE_SYSTEM);
  }
  STAMP(38);
  STAMP_END(1);
}

extern "C" void kernel_launch(int rank, int nranks, ncclComm_t comm, hipStream_t stream, void* const* d_in,
                              const int* in_sizes, int n_in, void* d_out, int out_size, const int* in_dtypes,
                              int out_dtype, void* const* d_out_all, void* const* d_ws_all, size_t ws_bytes) {
  if (nranks != NR || (unsigned)rank >= (unsigned)NR || n_in < 2 || ws_bytes < (size_t)WS_USED ||
      in_sizes[0] != SH * KD || in_sizes[1] != SH * KD || out_size != NR * SH * SH) {
    fprintf(stderr, "unexpected problem shape\n");
    return;
  }
  {
    int dev = -1, cnt = -1; char bus[64] = {0};
    (void)hipGetDevice(&dev); (void)hipGetDeviceCount(&cnt); (void)hipDeviceGetPCIBusId(bus, 64, dev);
    char line[512]; int n = snprintf(line, 512, "DIAG rank %d dev %d cnt %d bus %s peers", rank, dev, cnt, bus);
    for (int i = 0; i < NR; ++i) {
      hipPointerAttribute_t at; at.device = -9;
      hipError_t e = hipPointerGetAttributes(&at, d_ws_all[i]);
      n += snprintf(line + n, 512 - n, " %d/%d", (int)e, at.device);
    }
    (void)hipGetLastError();
    fprintf(stderr, "%s\n", line); fflush(stderr);
  }
  Params p;
  p.rank = 0;
  p.x = (const float*)d_in[0];
  p.w = (const float*)d_in[1];
  for (int i = 0; i < NR; ++i) {
    p.ws[i] = (char*)d_ws_all[phys(i)];
    p.out[i] = (char*)d_out_all[phys(i)];
    if (phys(i) == rank) p.rank = i;
  }
  HIP_CHECK(hipFuncSetAttribute((const void*)ag_gemm_kernel, hipFuncAttributeMaxDynamicSharedMemorySize, LDS_BYTES));
  ag_gemm_kernel<<<NWG, NTHR, LDS_BYTES, stream>>>(p);
  HIP_CHECK(hipGetLastError());
}
```

```cpp
#include <hip/hip_runtime.h>
#include <rccl/rccl.h>
#include <cstdio>

#define HIP_CHECK(expr) do { hipError_t _e = (expr); if (_e != hipSuccess) { \
    fprintf(stderr, "HIP error %s at %s:%d\n", hipGetErrorString(_e), __FILE__, __LINE__); \
    return; } } while (0)

typedef unsigned v4u __attribute__((ext_vector_type(4)));
typedef unsigned v2u __attribute__((ext_vector_type(2)));
typedef float f32x4 __attribute__((ext_vector_type(4)));
typedef float f32x2 __attribute__((ext_vector_type(2)));
typedef __bf16 bf16x2 __attribute__((ext_vector_type(2)));
typedef short bf16x8 __attribute__((ext_vector_type(8)));
typedef unsigned long long u64;
#define LAS __attribute__((address_space(3)))
#define GAS __attribute__((address_space(1)))

constexpr int NR = 8;
constexpr int KD = 4096;
constexpr int SH = 512;
constexpr int NWG = 256;
constexpr int NTHR = 512;
#define LINK_WGS 14
#define LINK_WAVES 1
constexpr unsigned PHYS = 0x75416320u;
__host__ __device__ constexpr int phys(int r) { return (PHYS >> (4 * r)) & 7; }
constexpr int NLW = LINK_WGS * LINK_WAVES;
constexpr int NSELF = (7 * (16 - LINK_WGS) + 16) * 8;
static_assert(LINK_WGS < 16 && 3 * NLW <= 128 * 5 && 3 * NSELF <= 128 * 13, "three fixed units per wave");
constexpr unsigned XS_OFF = 0;
constexpr unsigned WS_OFF = 16u << 20;
constexpr unsigned FLAG_OFF = 24u << 20;
constexpr unsigned FLAG_PITCH = 4096 + 512;
constexpr unsigned SLOT1_OFF = FLAG_OFF + (256u << 10);
constexpr unsigned SLOT1_PITCH = 1024 + 256;
constexpr unsigned SLOT2_OFF = SLOT1_OFF + (16u << 10);
constexpr unsigned DONE_OFF = SLOT2_OFF + (32u << 10);
constexpr unsigned CNT_OFF = DONE_OFF + (4u << 10);
constexpr unsigned TICK_OFF = CNT_OFF + (16u << 10);
constexpr unsigned TMO_OFF = TICK_OFF + (4u << 10);
constexpr unsigned WS_USED = TMO_OFF + 4096;
constexpr unsigned STAMP_OFF = 32u << 20;
constexpr u64 DEADLINE_TICKS = 50000000ull;
constexpr int CTL = 128 * 1024;
constexpr int LDS_BYTES = CTL + 256;
constexpr unsigned BAD_MAX = 0xfffffffeu;
#define AUX_DMA 17
constexpr int AUX_SYS = 17;

struct Params {
  const float* x;
  const float* w;
  char* ws[NR];
  char* out[NR];
  int rank;
};

#define RSRC(p, bytes) __builtin_amdgcn_make_buffer_rsrc((void*)(p), 0, (int)(bytes), 0x00020000)
#define CBAR() asm volatile("" ::: "memory")
#define WAITV(n) asm volatile("s_waitcnt vmcnt(" #n ")" ::: "memory")
#define WAITL() asm volatile("s_waitcnt lgkmcnt(0)" ::: "memory")
#define RFL(v) __builtin_amdgcn_readfirstlane(v)

#define STAMP(i) do {} while (0)
#define STAMP_END(cls) do {} while (0)

__device__ __forceinline__ unsigned pk(float a, float b) {
  f32x2 v = {a, b};
  return __builtin_bit_cast(unsigned, __builtin_convertvector(v, bf16x2));
}

__device__ __forceinline__ void unit_load(v4u (&r)[8], __amdgpu_buffer_rsrc_t rx, __amdgpu_buffer_rsrc_t rw,
                                          bool isw, int blk, int k32, int lane, unsigned oob) {
  const int kg = lane >> 4, c = lane & 15;
  if (!isw) {
    const unsigned base = ((unsigned)(blk * 64 + c) * (KD * 4) + (unsigned)(k32 * 32 + 4 * kg) * 4) | oob;
#pragma unroll
    for (int m = 0; m < 4; ++m) {
      r[2 * m] = __builtin_amdgcn_raw_buffer_load_b128(rx, base + m * (16 * KD * 4), 0, 0);
      r[2 * m + 1] = __builtin_amdgcn_raw_buffer_load_b128(rx, base + m * (16 * KD * 4) + 64, 0, 0);
    }
  } else {
    const unsigned base = ((unsigned)(k32 * 32 + 4 * kg) * (SH * 4) + (unsigned)(blk * 64 + 4 * c) * 4) | oob;
#pragma unroll
    for (int jj = 0; jj < 8; ++jj)
      r[jj] = __builtin_amdgcn_raw_buffer_load_b128(rw, base + ((jj & 3) + 16 * (jj >> 2)) * (SH * 4), 0, 0);
  }
}
__device__ __forceinline__ float asf(unsigned u) { return __builtin_bit_cast(float, u); }
__device__ __forceinline__ void unit_cvt(v4u (&o)[4], const v4u (&r)[8], bool isw) {
  if (!isw) {
#pragma unroll
    for (int m = 0; m < 4; ++m) {
      o[m].x = pk(asf(r[2 * m].x), asf(r[2 * m].y));
      o[m].y = pk(asf(r[2 * m].z), asf(r[2 * m].w));
      o[m].z = pk(asf(r[2 * m + 1].x), asf(r[2 * m + 1].y));
      o[m].w = pk(asf(r[2 * m + 1].z), asf(r[2 * m + 1].w));
    }
  } else {
#pragma unroll
    for (int i = 0; i < 4; ++i) {
      o[i].x = pk(asf(r[0][i]), asf(r[1][i]));
      o[i].y = pk(asf(r[2][i]), asf(r[3][i]));
      o[i].z = pk(asf(r[4][i]), asf(r[5][i]));
      o[i].w = pk(asf(r[6][i]), asf(r[7][i]));
    }
  }
}
__device__ __forceinline__ unsigned stage_off(int dblk, int k32) {
  const int kc = k32 >> 1, h = k32 & 1;
  return dblk < 32 ? XS_OFF + (unsigned)((kc * 32 + dblk) * 2 + h) * 4096u
                   : WS_OFF + (unsigned)((kc * 16 + (dblk - 32)) * 2 + h) * 4096u;
}
__device__ __forceinline__ unsigned flag_off(int dblk, int k32) {
  return FLAG_OFF + (unsigned)dblk * FLAG_PITCH + (unsigned)k32 * 4u;
}
__device__ __forceinline__ void unit_store(const v4u (&o)[4], __amdgpu_buffer_rsrc_t rd, unsigned off, int lane) {
#pragma unroll
  for (int i = 0; i < 4; ++i) __builtin_amdgcn_raw_buffer_store_b128(o[i], rd, off + lane * 16 + i * 1024, 0, AUX_SYS);
}

__device__ __forceinline__ bool spin_fail(u64& t0, unsigned* tmo, unsigned call) {
  const u64 now = __builtin_amdgcn_s_memrealtime();
  if (t0 == 0) t0 = now;
  if (now - t0 > 10000) {
    if (__hip_atomic_load(tmo, __ATOMIC_RELAXED, __HIP_MEMORY_SCOPE_AGENT) == call) return true;
    if (now - t0 > DEADLINE_TICKS) {
      __hip_atomic_store(tmo, call, __ATOMIC_RELAXED, __HIP_MEMORY_SCOPE_AGENT);
      return true;
    }
  }
  return false;
}

__device__ __forceinline__ unsigned wave_umax(unsigned v) {
  v = max(v, (unsigned)__builtin_amdgcn_update_dpp(0, (int)v, 0x111, 0xf, 0xf, true));
  v = max(v, (unsigned)__builtin_amdgcn_update_dpp(0, (int)v, 0x112, 0xf, 0xf, true));
  v = max(v, (unsigned)__builtin_amdgcn_update_dpp(0, (int)v, 0x114, 0xf, 0xf, true));
  v = max(v, (unsigned)__builtin_amdgcn_update_dpp(0, (int)v, 0x118, 0xf, 0xf, true));
  const unsigned a = __builtin_amdgcn_readlane(v, 15), b = __builtin_amdgcn_readlane(v, 31);
  const unsigned c = __builtin_amdgcn_readlane(v, 47), d = __builtin_amdgcn_readlane(v, 63);
  return max(max(a, b), max(c, d));
}

__global__ void __launch_bounds__(NTHR, 2) ag_gemm_kernel(Params p) {
  extern __shared__ __attribute__((aligned(16))) char smem[];
  LAS char* const L8 = (LAS char*)smem;
  const int lane = threadIdx.x & 63;
  const int wid = RFL(threadIdx.x >> 6);
  const int b = (blockIdx.x & ~7) | (__builtin_amdgcn_s_getreg((3 << 11) | 20) & 7);
  const int rank = p.rank, g = rank >> 2, jq = rank & 3;
  const int xcd = b & 7, bi = b >> 3;
  char* const myws = p.ws[rank];
  unsigned* const tmo = (unsigned*)(myws + TMO_OFF);
  unsigned* const cntp = (unsigned*)(myws + CNT_OFF + b * 64 + wid * 4);
  STAMP(0);

  if (bi < 16) {
    const bool link = xcd > 0 && bi < LINK_WGS;
    const int peer = ((g ^ (xcd >> 2)) << 2) | ((jq + xcd) & 3);
    if (link && wid >= LINK_WAVES) { STAMP_END(0); return; }
    const __amdgpu_buffer_rsrc_t rx = RSRC(p.x, SH * KD * 4);
    const __amdgpu_buffer_rsrc_t rw = RSRC(p.w, SH * KD * 4);
    const int dst = link ? peer : rank;
    const int mode = !link ? 2 : ((dst >> 2) == g ? 0 : 1);
    const int step = link ? NLW : NSELF;
    const int total = link ? 128 * 5 : 128 * 13;
    int u;
    if (link) u = wid * LINK_WGS + bi;
    else u = wid * (NSELF / 8) + (xcd == 0 ? bi : 16 + (xcd - 1) * (16 - LINK_WGS) + (bi - LINK_WGS));
    const __amdgpu_buffer_rsrc_t rd = RSRC(p.ws[dst], WS_USED);
    auto decode = [&](int uu, bool& isw, int& sblk, int& dblk, int& k32) {
      if (mode == 2) {
        k32 = uu / 13;
        const int s = uu - k32 * 13;
        isw = s >= 5; sblk = isw ? s - 5 : s; dblk = isw ? 32 + s - 5 : jq * 5 + s;
      } else {
        k32 = uu / 5;
        const int s = uu - k32 * 5;
        if (mode == 0) { isw = false; sblk = s; dblk = jq * 5 + s; }
        else if (s < 3) { isw = false; sblk = 5 + s; dblk = 20 + jq * 3 + s; }
        else { isw = true; sblk = (dst & 3) * 2 + (s - 3); dblk = 32 + 8 + jq * 2 + (s - 3); }
      }
    };
    v4u ra[8], rb2[8], o[4];
    bool isw, isw2; int sblk, dblk, k32, sblk2, dblk2, k322;
    decode(u, isw, sblk, dblk, k32);
    unit_load(ra, rx, rw, isw, sblk, k32, lane, 0);
    unsigned call;
    asm volatile("global_load_dword %0, %1, %2 sc1" : "=v"(call) : "v"(0), "s"(cntp) : "memory");
    unsigned pflag = 0xffffffffu;
    const unsigned lmask = lane == 0 ? 0u : 0xffffffffu;
    const __amdgpu_buffer_rsrc_t rs = RSRC(myws, WS_USED);
    const unsigned tko = TICK_OFF + (link ? xcd : 0) * 256;
    unsigned tk = 1, tbase = 0;
    int stat = u + 2 * step;
    u += step;
    int t = 0;
#define DRAW() do { u64 sv_; tk = 1; asm volatile("s_mov_b64 %1, exec\n\ts_cmp_lt_u32 %4, %5\n\ts_cselect_b64 exec, 1, 0\n\tbuffer_atomic_add %0, %2, %3, 0 offen sc0 sc1\n\ts_mov_b64 exec, %1" : "+v"(tk), "=&s"(sv_) : "v"(tko), "s"(rs), "s"(u), "s"(total) : "memory", "scc"); } while (0)
#pragma unroll 1
    for (;;) {
      bool more = (unsigned)u < (unsigned)total;
      decode(u, isw2, sblk2, dblk2, k322);
      unit_load(rb2, rx, rw, isw2, sblk2, k322, lane, more ? 0u : 0x80000000u);
      WAITV(8);
      asm volatile("" : "+v"(call), "+v"(tk));
      tbase = (unsigned)RFL(call) * (unsigned)(total - 2 * step) - 3u * step;
      u = stat >= 0 ? stat : (int)((unsigned)RFL(tk) - tbase);
      stat = -1;
      STAMP(1 + t);
      __builtin_amdgcn_raw_buffer_store_b32(call + 1, rd, pflag | lmask, 0, AUX_SYS);
      unit_cvt(o, ra, isw);
      unit_store(o, rd, stage_off(dblk, k32), lane);
      pflag = flag_off(dblk, k32);
      if (!more) break;
      more = (unsigned)u < (unsigned)total;
      DRAW();
      decode(u, isw, sblk, dblk, k32);
      unit_load(ra, rx, rw, isw, sblk, k32, lane, more ? 0u : 0x80000000u);
      WAITV(8);
      asm volatile("" : "+v"(tk));
      u = (int)((unsigned)RFL(tk) - tbase);
      STAMP(2 + t);
      __builtin_amdgcn_raw_buffer_store_b32(call + 1, rd, pflag | lmask, 0, AUX_SYS);
      unit_cvt(o, rb2, isw2);
      unit_store(o, rd, stage_off(dblk2, k322), lane);
      pflag = flag_off(dblk2, k322);
      if (!more) break;
      DRAW();
      t = min(t + 2, 56);
    }
    call += 1;
    WAITV(0);
    STAMP(60);
    if (lane == 0) {
      __builtin_amdgcn_raw_buffer_store_b32(call, rd, pflag, 0, AUX_SYS);
      __hip_atomic_store(cntp, call, __ATOMIC_RELAXED, __HIP_MEMORY_SCOPE_AGENT);
    }
    if (bi == 15 && (wid < 2 || (b == 127 && wid >= 4))) {
      u64 t0 = 0;
      if (wid < 2) {
        const __amdgpu_buffer_rsrc_t rq = RSRC(p.ws[peer], WS_USED);
        unsigned m;
        for (;;) {
          const v4u a = __builtin_amdgcn_raw_buffer_load_b128(rs, SLOT1_OFF + xcd * SLOT1_PITCH + lane * 16, 0, 16);
          CBAR();
          m = max(a.x, a.z);
          if (__all(a.y == call && a.w == call)) break;
          if (spin_fail(t0, tmo, call)) { m = BAD_MAX; break; }
        }
        const v2u mv = {wave_umax(m), call};
        __builtin_amdgcn_raw_buffer_store_b64(mv, rq, SLOT2_OFF + lane * 256 + rank * 8, 0, AUX_SYS);
        __builtin_amdgcn_raw_buffer_store_b64(mv, rq, SLOT2_OFF + (lane + 64) * 256 + rank * 8, 0, AUX_SYS);
        STAMP(61);
      } else {
        bool bad = false;
        for (;;) {
          const unsigned a = __builtin_amdgcn_raw_buffer_load_b32(rs, DONE_OFF + (wid - 4) * 256 + lane * 4, 0, AUX_SYS);
          CBAR();
          if (__all(a >= call)) break;
          if (spin_fail(t0, tmo, call)) { bad = true; break; }
        }
        if (bad && lane == 0) {
          __builtin_amdgcn_s_sleep(127);
          p.out[rank][(size_t)phys(rank) * SH * SH] = (char)-128;
        }
        STAMP(62);
      }
      STAMP_END(3);
      return;
    }
    STAMP_END(link ? 2 : 5);
    return;
  }

  volatile LAS unsigned* const ctl = (volatile LAS unsigned*)(L8 + CTL);
  const unsigned call = __hip_atomic_load(cntp, __ATOMIC_RELAXED, __HIP_MEMORY_SCOPE_AGENT) + 1;
  if (threadIdx.x < 16) ctl[threadIdx.x] = 0;
  if (lane == 0) __hip_atomic_store(cntp, call, __ATOMIC_RELAXED, __HIP_MEMORY_SCOPE_AGENT);
  WAITL();
  __builtin_amdgcn_s_barrier();
  CBAR();
  const int ti = bi - 16;
  const int rbb = (xcd >> 1) * 4 + (ti & 3);
  const int cti = ti >> 2;
  const int ct = (cti & 1) + 2 * (xcd & 1) + 4 * (cti >> 1);
  const __amdgpu_buffer_rsrc_t rs = RSRC(myws, WS_USED);

  if (wid >= 4) {
    const int s = wid - 4;
    const int blk = s < 2 ? 2 * rbb + s : 32 + 2 * ct + (s - 2);
    const unsigned sbase = (s < 2 ? XS_OFF + blk * 8192 : WS_OFF + (blk - 32) * 8192) + lane * 16;
    const unsigned sstep = s < 2 ? 32 * 8192 : 16 * 8192;
    const int c0 = s < 2 ? 2 * s : s - 2, c1 = s < 2 ? 2 * s + 1 : s;
    const unsigned fo = FLAG_OFF + blk * FLAG_PITCH + lane * 8;
    int avail = 0, issued = 0, pub = 0;
    u64 t0 = 0;
#pragma unroll 1
    while (pub < 64) {
      const int fr = RFL(min(ctl[4 + c0], ctl[4 + c1])) + 4;
      const int lim = min(avail, fr);
      CBAR();
#pragma unroll 1
      while (issued < lim) {
        LAS char* dl = L8 + s * 32768 + (issued & 3) * 8192;
#pragma unroll
        for (int i = 0; i < 8; ++i)
          __builtin_amdgcn_raw_ptr_buffer_load_lds(rs, (LAS void*)(dl + i * 1024), 16, sbase + i * 1024, issued * sstep, 0, AUX_DMA);
        ++issued;
      }
      if (issued == avail && avail < 64) {
        v2u f = {0u, 0u};
        const bool inw = (unsigned)(lane - avail) < 16u;
        if (inw) f = __builtin_amdgcn_raw_buffer_load_b64(rs, fo, 0, AUX_SYS);
        const int o = issued - pub;
        if (o >= 4) { WAITV(25); ctl[s] = issued - 3; }
        if (o >= 3) { WAITV(17); ctl[s] = issued - 2; }
        if (o >= 2) { WAITV(9); ctl[s] = issued - 1; }
        if (o >= 1) { WAITV(1); ctl[s] = issued; STAMP(1 + (issued >> 1)); }
        pub = issued;
        CBAR();
        const u64 mk = __ballot(lane < avail || (inw && f.x >= call && f.y >= call));
        const int na = ~mk ? __builtin_ctzll(~mk) : 64;
        if (na == avail) {
          if (spin_fail(t0, tmo, call)) { avail = 64; ctl[12] = 1; }
        } else {
          avail = na; t0 = 0;
        }
      } else if (pub < issued) {
        const int o = issued - pub;
        if (o >= 4) WAITV(24);
        else if (o == 3) WAITV(16);
        else if (o == 2) WAITV(8);
        else WAITV(0);
        ++pub;
        ctl[s] = pub;
        STAMP(1 + (pub >> 1));
        t0 = 0;
      } else {
        if (spin_fail(t0, tmo, call)) { ctl[12] = 1; break; }
      }
    }
    WAITV(0);
    WAITL();
    STAMP_END(4);
    return;
  }

  const int wm = wid >> 1, wn = wid & 1;
  const bool remote = ct >= 4;
  const int orank = (remote ? (1 - g) * 4 + (ct - 4) : rank) & 7;
  u64 donep = (u64)p.ws[orank] + DONE_OFF + ((jq * 16 + rbb) * 4 + wid) * 4;
  unsigned outlo = (unsigned)(u64)p.out[orank], outhi = (unsigned)((u64)p.out[orank] >> 32);
  asm volatile("" : "+v"(donep), "+s"(outlo), "+s"(outhi));
  char* const outp = (char*)(((u64)(unsigned)RFL(outhi) << 32) | (u64)(unsigned)RFL(outlo));
  f32x4 acc[4][4];
#pragma unroll
  for (int m = 0; m < 4; ++m)
#pragma unroll
    for (int n = 0; n < 4; ++n) acc[m][n] = f32x4{0.f, 0.f, 0.f, 0.f};
  {
    const LAS char* xb = L8 + wm * 32768 + lane * 16;
    const LAS char* wb = L8 + (2 + wn) * 32768 + lane * 16;
    int full = 0;
    v4u xa[4], wa[4], xc[4], wc[4];
    auto rd_half = [&](v4u (&xf)[4], v4u (&wf)[4], int off) {
#pragma unroll
      for (int m = 0; m < 4; ++m) wf[m] = *(const LAS v4u*)(wb + off + m * 1024);
#pragma unroll
      for (int m = 0; m < 4; ++m) xf[m] = *(const LAS v4u*)(xb + off + m * 1024);
    };
    auto mm = [&](const v4u (&xf)[4], const v4u (&wf)[4]) {
#pragma unroll
      for (int m = 0; m < 4; ++m)
#pragma unroll
        for (int n = 0; n < 4; ++n)
          acc[m][n] = __builtin_amdgcn_mfma_f32_16x16x32_bf16(__builtin_bit_cast(bf16x8, wf[n]),
                                                              __builtin_bit_cast(bf16x8, xf[m]), acc[m][n], 0, 0, 0);
    };
    auto wait_full = [&](int kc) {
      u64 t0 = 0;
      int sp = 0;
      while (full <= kc) {
        full = RFL(min(ctl[wm], ctl[2 + wn]));
        if ((++sp & 63) == 0 && spin_fail(t0, tmo, call)) { full = 64; ctl[12] = 1; }
      }
      CBAR();
    };
    wait_full(0);
    rd_half(xa, wa, 0);
#pragma unroll 1
    for (int kc = 0; kc < 64; ++kc) {
      if ((kc & 1) == 0) STAMP(1 + (kc >> 1));
      rd_half(xc, wc, (kc & 3) * 8192 + 4096);
      const unsigned f0 = ctl[wm], f1 = ctl[2 + wn];
      __builtin_amdgcn_sched_barrier(0);
      mm(xa, wa);
      __builtin_amdgcn_sched_barrier(0);
      WAITL();
      ctl[4 + wid] = kc + 1;
      full = max(full, RFL(min(f0, f1)));
      const bool ahead = full > kc + 1;
      if (ahead) rd_half(xa, wa, ((kc + 1) & 3) * 8192);
      __builtin_amdgcn_sched_barrier(0);
      mm(xc, wc);
      __builtin_amdgcn_sched_barrier(0);
      if (!ahead && kc < 63) {
        wait_full(kc + 1);
        rd_half(xa, wa, ((kc + 1) & 3) * 8192);
      }
    }
  }
  STAMP(34);
  float mx = 0.f;
#pragma unroll
  for (int m = 0; m < 4; ++m)
#pragma unroll
    for (int n = 0; n < 4; ++n)
#pragma unroll
      for (int e = 0; e < 4; ++e) mx = fmaxf(mx, acc[m][n][e]);
  bool bad;
  {
    const unsigned wmx = wave_umax(__builtin_bit_cast(unsigned, mx));
    if (lane == 0) ctl[8 + wid] = wmx;
    WAITL();
    __builtin_amdgcn_s_barrier();
    CBAR();
    if (wid == 0 && lane < 8) {
      const unsigned m4 = max(max(ctl[8], ctl[9]), max(ctl[10], ctl[11]));
      const v2u mv = {m4, call};
      __builtin_amdgcn_raw_buffer_store_b64(mv, rs, SLOT1_OFF + lane * SLOT1_PITCH + (b - 128) * 8, 0, 16);
    }
    bad = ctl[12] != 0;
  }
  STAMP(35);
#pragma unroll
  for (int m = 0; m < 4; ++m)
#pragma unroll
    for (int n = 0; n < 4; ++n)
#pragma unroll
      for (int e = 0; e < 4; ++e) acc[m][n][e] = fmaxf(acc[m][n][e], 0.f);
#pragma unroll
  for (int m = 0; m < 4; ++m) asm volatile("" : "+v"(acc[m][0]), "+v"(acc[m][1]), "+v"(acc[m][2]), "+v"(acc[m][3]));
  unsigned gm;
  {
    const unsigned so = SLOT2_OFF + (b - 128) * 256 + (lane & 7) * 8;
    u64 t0 = 0;
    for (int sp = 1;; ++sp) {
      if (wid < 2) {
        const v2u a = __builtin_amdgcn_raw_buffer_load_b64(rs, so, 0, AUX_SYS);
        CBAR();
        if (__all(a.y >= call)) {
          gm = a.x;
          gm = max(gm, (unsigned)__builtin_amdgcn_update_dpp(0, (int)gm, 0x111, 0xf, 0xf, true));
          gm = max(gm, (unsigned)__builtin_amdgcn_update_dpp(0, (int)gm, 0x112, 0xf, 0xf, true));
          gm = max(gm, (unsigned)__builtin_amdgcn_update_dpp(0, (int)gm, 0x114, 0xf, 0xf, true));
          gm = __builtin_amdgcn_readlane(gm, 7);
          ctl[13] = gm + 1;
          break;
        }
      }
      gm = RFL(ctl[13]) - 1;
      if (gm != 0xffffffffu) break;
      if ((wid < 2 || (sp & 63) == 0) && spin_fail(t0, tmo, call)) { gm = BAD_MAX; break; }
    }
  }
  STAMP(36);
  bad = bad || gm == BAD_MAX;
  const float amax = __builtin_bit_cast(float, gm);
  const float inv = amax > 0.f && !bad ? 127.0f * __builtin_amdgcn_rcpf(amax) : 0.f;
  const float magic = bad ? 12582912.0f + 128.0f : 12582912.0f;
  const int rb = 2 * rbb + wm;
  int srank, lrow;
  if (rb < 20) { srank = g * 4 + rb / 5; lrow = (rb % 5) * 64; }
  else { srank = (1 - g) * 4 + (rb - 20) / 3; lrow = (5 + (rb - 20) % 3) * 64; }
  const int ocol = (remote ? jq * 128 : ct * 128) + wn * 64;
  const __amdgpu_buffer_rsrc_t ro = RSRC(outp, NR * SH * SH);
  const unsigned obase = (unsigned)(phys(srank) * SH + lrow + (lane & 15)) * SH + ocol + (lane >> 4) * 16;
#pragma unroll
  for (int m = 0; m < 4; ++m) {
    v4u q;
    const f32x4 inv4 = {inv, inv, inv, inv}, mg4 = {magic, magic, magic, magic};
    v4u tb[4];
#pragma unroll
    for (int n = 0; n < 4; ++n) tb[n] = __builtin_bit_cast(v4u, __builtin_elementwise_fma(acc[m][n], inv4, mg4));
#pragma unroll
    for (int e = 0; e < 4; ++e) {
      const unsigned lo = __builtin_amdgcn_perm(tb[1][e], tb[0][e], 0x0c0c0400u);
      const unsigned hi = __builtin_amdgcn_perm(tb[3][e], tb[2][e], 0x04000c0cu);
      q[e] = lo | hi;
    }
    __builtin_amdgcn_raw_buffer_store_b128(q, ro, obase + m * (16 * SH), 0, AUX_SYS);
  }
  STAMP(37);
  if (remote) {
    WAITV(0);
    if (lane == 0)
      __hip_atomic_store((GAS unsigned*)donep, call, __ATOMIC_RELAXED, __HIP_MEMORY_SCOPE_SYSTEM);
  }
  STAMP(38);
  STAMP_END(1);
}

extern "C" void kernel_launch(int rank, int nranks, ncclComm_t comm, hipStream_t stream, void* const* d_in,
                              const int* in_sizes, int n_in, void* d_out, int out_size, const int* in_dtypes,
                              int out_dtype, void* const* d_out_all, void* const* d_ws_all, size_t ws_bytes) {
  if (nranks != NR || (unsigned)rank >= (unsigned)NR || n_in < 2 || ws_bytes < (size_t)WS_USED ||
      in_sizes[0] != SH * KD || in_sizes[1] != SH * KD || out_size != NR * SH * SH) {
    fprintf(stderr, "unexpected problem shape\n");
    return;
  }
  Params p;
  p.rank = 0;
  p.x = (const float*)d_in[0];
  p.w = (const float*)d_in[1];
  for (int i = 0; i < NR; ++i) {
    p.ws[i] = (char*)d_ws_all[phys(i)];
    p.out[i] = (char*)d_out_all[phys(i)];
    if (phys(i) == rank) p.rank = i;
  }
  HIP_CHECK(hipFuncSetAttribute((const void*)ag_gemm_kernel, hipFuncAttributeMaxDynamicSharedMemorySize, LDS_BYTES));
  ag_gemm_kernel<<<NWG, NTHR, LDS_BYTES, stream>>>(p);
  HIP_CHECK(hipGetLastError());
}
```

```cpp
#include <hip/hip_runtime.h>
#include <rccl/rccl.h>
#include <cstdio>

#define HIP_CHECK(expr) do { hipError_t _e = (expr); if (_e != hipSuccess) { \
    fprintf(stderr, "HIP error %s at %s:%d\n", hipGetErrorString(_e), __FILE__, __LINE__); \
    return; } } while (0)

typedef unsigned v4u __attribute__((ext_vector_type(4)));
typedef unsigned v2u __attribute__((ext_vector_type(2)));
typedef float f32x4 __attribute__((ext_vector_type(4)));
typedef float f32x2 __attribute__((ext_vector_type(2)));
typedef __bf16 bf16x2 __attribute__((ext_vector_type(2)));
typedef short bf16x8 __attribute__((ext_vector_type(8)));
typedef unsigned long long u64;
#define LAS __attribute__((address_space(3)))
#define GAS __attribute__((address_space(1)))

constexpr int NR = 8;
constexpr int KD = 4096;
constexpr int SH = 512;
constexpr int NWG = 256;
constexpr int NTHR = 512;
#define LINK_WGS 14
#define LINK_WAVES 1
constexpr unsigned PHYS = 0x75416320u;
__host__ __device__ constexpr int phys(int r) { return (PHYS >> (4 * r)) & 7; }
constexpr int NLW = LINK_WGS * LINK_WAVES;
constexpr int NSELF = (7 * (16 - LINK_WGS) + 16) * 8;
static_assert(LINK_WGS < 16 && 3 * NLW <= 128 * 5 && 3 * NSELF <= 128 * 13, "three fixed units per wave");
constexpr unsigned XS_OFF = 0;
constexpr unsigned WS_OFF = 16u << 20;
constexpr unsigned FLAG_OFF = 24u << 20;
constexpr unsigned FLAG_PITCH = 4096 + 512;
constexpr unsigned SLOT1_OFF = FLAG_OFF + (256u << 10);
constexpr unsigned SLOT1_PITCH = 1024 + 256;
constexpr unsigned SLOT2_OFF = SLOT1_OFF + (16u << 10);
constexpr unsigned DONE_OFF = SLOT2_OFF + (32u << 10);
constexpr unsigned CNT_OFF = DONE_OFF + (4u << 10);
constexpr unsigned TICK_OFF = CNT_OFF + (16u << 10);
constexpr unsigned TMO_OFF = TICK_OFF + (4u << 10);
constexpr unsigned WS_USED = TMO_OFF + 4096;
constexpr unsigned STAMP_OFF = 32u << 20;
constexpr u64 DEADLINE_TICKS = 50000000ull;
constexpr int CTL = 128 * 1024;
constexpr int LDS_BYTES = CTL + 256;
constexpr unsigned BAD_MAX = 0xfffffffeu;
#define AUX_DMA 17
constexpr int AUX_SYS = 17;

struct Params {
  const float* x;
  const float* w;
  char* myws;
  int rank, pad;
  char* ws[NR];
  char* out[NR];
};

#define RSRC(p, bytes) __builtin_amdgcn_make_buffer_rsrc((void*)(p), 0, (int)(bytes), 0x00020000)
#define CBAR() asm volatile("" ::: "memory")
#define WAITV(n) asm volatile("s_waitcnt vmcnt(" #n ")" ::: "memory")
#define WAITL() asm volatile("s_waitcnt lgkmcnt(0)" ::: "memory")
#define RFL(v) __builtin_amdgcn_readfirstlane(v)

#define STAMP(i) do {} while (0)
#define STAMP_END(cls) do {} while (0)

__device__ __forceinline__ unsigned pk(float a, float b) {
  f32x2 v = {a, b};
  return __builtin_bit_cast(unsigned, __builtin_convertvector(v, bf16x2));
}

__device__ __forceinline__ void unit_load(v4u (&r)[8], __amdgpu_buffer_rsrc_t rx, __amdgpu_buffer_rsrc_t rw,
                                          bool isw, int blk, int k32, int lane, unsigned oob) {
  const int kg = lane >> 4, c = lane & 15;
  if (!isw) {
    const unsigned base = ((unsigned)(blk * 64 + c) * (KD * 4) + (unsigned)(k32 * 32 + 4 * kg) * 4) | oob;
#pragma unroll
    for (int m = 0; m < 4; ++m) {
      r[2 * m] = __builtin_amdgcn_raw_buffer_load_b128(rx, base + m * (16 * KD * 4), 0, 0);
      r[2 * m + 1] = __builtin_amdgcn_raw_buffer_load_b128(rx, base + m * (16 * KD * 4) + 64, 0, 0);
    }
  } else {
    const unsigned base = ((unsigned)(k32 * 32 + 4 * kg) * (SH * 4) + (unsigned)(blk * 64 + 4 * c) * 4) | oob;
#pragma unroll
    for (int jj = 0; jj < 8; ++jj)
      r[jj] = __builtin_amdgcn_raw_buffer_load_b128(rw, base + ((jj & 3) + 16 * (jj >> 2)) * (SH * 4), 0, 0);
  }
}
__device__ __forceinline__ float asf(unsigned u) { return __builtin_bit_cast(float, u); }
__device__ __forceinline__ void unit_cvt(v4u (&o)[4], const v4u (&r)[8], bool isw) {
  if (!isw) {
#pragma unroll
    for (int m = 0; m < 4; ++m) {
      o[m].x = pk(asf(r[2 * m].x), asf(r[2 * m].y));
      o[m].y = pk(asf(r[2 * m].z), asf(r[2 * m].w));
      o[m].z = pk(asf(r[2 * m + 1].x), asf(r[2 * m + 1].y));
      o[m].w = pk(asf(r[2 * m + 1].z), asf(r[2 * m + 1].w));
    }
  } else {
#pragma unroll
    for (int i = 0; i < 4; ++i) {
      o[i].x = pk(asf(r[0][i]), asf(r[1][i]));
      o[i].y = pk(asf(r[2][i]), asf(r[3][i]));
      o[i].z = pk(asf(r[4][i]), asf(r[5][i]));
      o[i].w = pk(asf(r[6][i]), asf(r[7][i]));
    }
  }
}
__device__ __forceinline__ unsigned stage_off(int dblk, int k32) {
  const int kc = k32 >> 1, h = k32 & 1;
  return dblk < 32 ? XS_OFF + (unsigned)((kc * 32 + dblk) * 2 + h) * 4096u
                   : WS_OFF + (unsigned)((kc * 16 + (dblk - 32)) * 2 + h) * 4096u;
}
__device__ __forceinline__ unsigned flag_off(int dblk, int k32) {
  return FLAG_OFF + (unsigned)dblk * FLAG_PITCH + (unsigned)k32 * 4u;
}
__device__ __forceinline__ void unit_store(const v4u (&o)[4], __amdgpu_buffer_rsrc_t rd, unsigned off, int lane) {
#pragma unroll
  for (int i = 0; i < 4; ++i) __builtin_amdgcn_raw_buffer_store_b128(o[i], rd, off + lane * 16 + i * 1024, 0, AUX_SYS);
}

__device__ __forceinline__ bool spin_fail(u64& t0, unsigned* tmo, unsigned call) {
  const u64 now = __builtin_amdgcn_s_memrealtime();
  if (t0 == 0) t0 = now;
  if (now - t0 > 10000) {
    if (__hip_atomic_load(tmo, __ATOMIC_RELAXED, __HIP_MEMORY_SCOPE_AGENT) == call) return true;
    if (now - t0 > DEADLINE_TICKS) {
      __hip_atomic_store(tmo, call, __ATOMIC_RELAXED, __HIP_MEMORY_SCOPE_AGENT);
      return true;
    }
  }
  return false;
}

__device__ __forceinline__ unsigned wave_umax(unsigned v) {
  v = max(v, (unsigned)__builtin_amdgcn_update_dpp(0, (int)v, 0x111, 0xf, 0xf, true));
  v = max(v, (unsigned)__builtin_amdgcn_update_dpp(0, (int)v, 0x112, 0xf, 0xf, true));
  v = max(v, (unsigned)__builtin_amdgcn_update_dpp(0, (int)v, 0x114, 0xf, 0xf, true));
  v = max(v, (unsigned)__builtin_amdgcn_update_dpp(0, (int)v, 0x118, 0xf, 0xf, true));
  const unsigned a = __builtin_amdgcn_readlane(v, 15), b = __builtin_amdgcn_readlane(v, 31);
  const unsigned c = __builtin_amdgcn_readlane(v, 47), d = __builtin_amdgcn_readlane(v, 63);
  return max(max(a, b), max(c, d));
}

__global__ void __launch_bounds__(NTHR, 2) ag_gemm_kernel(Params p) {
  extern __shared__ __attribute__((aligned(16))) char smem[];
  LAS char* const L8 = (LAS char*)smem;
  const int lane = threadIdx.x & 63;
  const int wid = RFL(threadIdx.x >> 6);
  const int b = (blockIdx.x & ~7) | (__builtin_amdgcn_s_getreg((3 << 11) | 20) & 7);
  const float* px = p.x;
  const float* pw = p.w;
  u64 wsv = (u64)p.myws;
  int rank = p.rank;
  asm volatile("" : "+s"(px), "+s"(pw), "+s"(wsv), "+s"(rank));
  char* const myws = (char*)(GAS char*)wsv;
  const int g = rank >> 2, jq = rank & 3;
  const int xcd = b & 7, bi = b >> 3;
  unsigned* const tmo = (unsigned*)(myws + TMO_OFF);
  unsigned* const cntp = (unsigned*)(myws + CNT_OFF + b * 64 + wid * 4);
  STAMP(0);

  if (__builtin_expect(bi < 16, 1)) {
    const bool link = xcd > 0 && bi < LINK_WGS;
    const int peer = ((g ^ (xcd >> 2)) << 2) | ((jq + xcd) & 3);
    if (link && wid >= LINK_WAVES) { STAMP_END(0); return; }
    const __amdgpu_buffer_rsrc_t rx = RSRC(px, SH * KD * 4);
    const __amdgpu_buffer_rsrc_t rw = RSRC(pw, SH * KD * 4);
    const int dst = link ? peer : rank;
    const int mode = !link ? 2 : ((dst >> 2) == g ? 0 : 1);
    const int step = link ? NLW : NSELF;
    const int total = link ? 128 * 5 : 128 * 13;
    int u;
    if (link) u = wid * LINK_WGS + bi;
    else u = wid * (NSELF / 8) + (xcd == 0 ? bi : 16 + (xcd - 1) * (16 - LINK_WGS) + (bi - LINK_WGS));
    const __amdgpu_buffer_rsrc_t rd = RSRC(p.ws[dst], WS_USED);
    auto decode = [&](int uu, bool& isw, int& sblk, int& dblk, int& k32) {
      if (mode == 2) {
        k32 = uu / 13;
        const int s = uu - k32 * 13;
        isw = s >= 5; sblk = isw ? s - 5 : s; dblk = isw ? 32 + s - 5 : jq * 5 + s;
      } else {
        k32 = uu / 5;
        const int s = uu - k32 * 5;
        if (mode == 0) { isw = false; sblk = s; dblk = jq * 5 + s; }
        else if (s < 3) { isw = false; sblk = 5 + s; dblk = 20 + jq * 3 + s; }
        else { isw = true; sblk = (dst & 3) * 2 + (s - 3); dblk = 32 + 8 + jq * 2 + (s - 3); }
      }
    };
    v4u ra[8], rb2[8], o[4];
    bool isw, isw2; int sblk, dblk, k32, sblk2, dblk2, k322;
    decode(u, isw, sblk, dblk, k32);
    unit_load(ra, rx, rw, isw, sblk, k32, lane, 0);
    unsigned call;
    asm volatile("global_load_dword %0, %1, %2 sc1" : "=v"(call) : "v"(0), "s"(cntp) : "memory");
    unsigned pflag = 0xffffffffu;
    const unsigned lmask = lane == 0 ? 0u : 0xffffffffu;
    const __amdgpu_buffer_rsrc_t rs = RSRC(myws, WS_USED);
    const unsigned tko = TICK_OFF + (link ? xcd : 0) * 256;
    unsigned tk = 1, tbase = 0;
    int stat = u + 2 * step;
    u += step;
    int t = 0;
#define DRAW() do { u64 sv_; tk = 1; asm volatile("s_mov_b64 %1, exec\n\ts_cmp_lt_u32 %4, %5\n\ts_cselect_b64 exec, 1, 0\n\tbuffer_atomic_add %0, %2, %3, 0 offen sc0 sc1\n\ts_mov_b64 exec, %1" : "+v"(tk), "=&s"(sv_) : "v"(tko), "s"(rs), "s"(u), "s"(total) : "memory", "scc"); } while (0)
#pragma unroll 1
    for (;;) {
      bool more = (unsigned)u < (unsigned)total;
      decode(u, isw2, sblk2, dblk2, k322);
      unit_load(rb2, rx, rw, isw2, sblk2, k322, lane, more ? 0u : 0x80000000u);
      WAITV(8);
      asm volatile("" : "+v"(call), "+v"(tk));
      tbase = (unsigned)RFL(call) * (unsigned)(total - 2 * step) - 3u * step;
      u = stat >= 0 ? stat : (int)((unsigned)RFL(tk) - tbase);
      stat = -1;
      STAMP(1 + t);
      __builtin_amdgcn_raw_buffer_store_b32(call + 1, rd, pflag | lmask, 0, AUX_SYS);
      unit_cvt(o, ra, isw);
      unit_store(o, rd, stage_off(dblk, k32), lane);
      pflag = flag_off(dblk, k32);
      if (!more) break;
      more = (unsigned)u < (unsigned)total;
      DRAW();
      decode(u, isw, sblk, dblk, k32);
      unit_load(ra, rx, rw, isw, sblk, k32, lane, more ? 0u : 0x80000000u);
      WAITV(8);
      asm volatile("" : "+v"(tk));
      u = (int)((unsigned)RFL(tk) - tbase);
      STAMP(2 + t);
      __builtin_amdgcn_raw_buffer_store_b32(call + 1, rd, pflag | lmask, 0, AUX_SYS);
      unit_cvt(o, rb2, isw2);
      unit_store(o, rd, stage_off(dblk2, k322), lane);
      pflag = flag_off(dblk2, k322);
      if (!more) break;
      DRAW();
      t = min(t + 2, 56);
    }
    call += 1;
    WAITV(0);
    STAMP(60);
    if (lane == 0) {
      __builtin_amdgcn_raw_buffer_store_b32(call, rd, pflag, 0, AUX_SYS);
      __hip_atomic_store(cntp, call, __ATOMIC_RELAXED, __HIP_MEMORY_SCOPE_AGENT);
    }
    if (bi == 15 && (wid < 2 || (b == 127 && wid >= 4))) {
      u64 t0 = 0;
      if (wid < 2) {
        const __amdgpu_buffer_rsrc_t rq = RSRC(p.ws[peer], WS_USED);
        unsigned m;
        for (;;) {
          const v4u a = __builtin_amdgcn_raw_buffer_load_b128(rs, SLOT1_OFF + xcd * SLOT1_PITCH + lane * 16, 0, 16);
          CBAR();
          m = max(a.x, a.z);
          if (__all(a.y == call && a.w == call)) break;
          if (spin_fail(t0, tmo, call)) { m = BAD_MAX; break; }
        }
        const v2u mv = {wave_umax(m), call};
        __builtin_amdgcn_raw_buffer_store_b64(mv, rq, SLOT2_OFF + lane * 256 + rank * 8, 0, AUX_SYS);
        __builtin_amdgcn_raw_buffer_store_b64(mv, rq, SLOT2_OFF + (lane + 64) * 256 + rank * 8, 0, AUX_SYS);
        STAMP(61);
      } else {
        bool bad = false;
        for (;;) {
          const unsigned a = __builtin_amdgcn_raw_buffer_load_b32(rs, DONE_OFF + (wid - 4) * 256 + lane * 4, 0, AUX_SYS);
          CBAR();
          if (__all(a >= call)) break;
          if (spin_fail(t0, tmo, call)) { bad = true; break; }
        }
        if (bad && lane == 0) {
          __builtin_amdgcn_s_sleep(127);
          p.out[rank][(size_t)phys(rank) * SH * SH] = (char)-128;
        }
        STAMP(62);
      }
      STAMP_END(3);
      return;
    }
    STAMP_END(link ? 2 : 5);
    return;
  }

  volatile LAS unsigned* const ctl = (volatile LAS unsigned*)(L8 + CTL);
  const unsigned call = __hip_atomic_load(cntp, __ATOMIC_RELAXED, __HIP_MEMORY_SCOPE_AGENT) + 1;
  if (threadIdx.x < 16) ctl[threadIdx.x] = 0;
  if (lane == 0) __hip_atomic_store(cntp, call, __ATOMIC_RELAXED, __HIP_MEMORY_SCOPE_AGENT);
  WAITL();
  __builtin_amdgcn_s_barrier();
  CBAR();
  const int ti = bi - 16;
  const int rbb = (xcd >> 1) * 4 + (ti & 3);
  const int cti = ti >> 2;
  const int ct = (cti & 1) + 2 * (xcd & 1) + 4 * (cti >> 1);
  const __amdgpu_buffer_rsrc_t rs = RSRC(myws, WS_USED);

  if (wid >= 4) {
    const int s = wid - 4;
    const int blk = s < 2 ? 2 * rbb + s : 32 + 2 * ct + (s - 2);
    const unsigned sbase = (s < 2 ? XS_OFF + blk * 8192 : WS_OFF + (blk - 32) * 8192) + lane * 16;
    const unsigned sstep = s < 2 ? 32 * 8192 : 16 * 8192;
    const int c0 = s < 2 ? 2 * s : s - 2, c1 = s < 2 ? 2 * s + 1 : s;
    const unsigned fo = FLAG_OFF + blk * FLAG_PITCH + lane * 8;
    int avail = 0, issued = 0, pub = 0;
    u64 t0 = 0;
#pragma unroll 1
    while (pub < 64) {
      const int fr = RFL(min(ctl[4 + c0], ctl[4 + c1])) + 4;
      const int lim = min(avail, fr);
      CBAR();
#pragma unroll 1
      while (issued < lim) {
        LAS char* dl = L8 + s * 32768 + (issued & 3) * 8192;
#pragma unroll
        for (int i = 0; i < 8; ++i)
          __builtin_amdgcn_raw_ptr_buffer_load_lds(rs, (LAS void*)(dl + i * 1024), 16, sbase + i * 1024, issued * sstep, 0, AUX_DMA);
        ++issued;
      }
      if (issued == avail && avail < 64) {
        v2u f = {0u, 0u};
        const bool inw = (unsigned)(lane - avail) < 16u;
        if (inw) f = __builtin_amdgcn_raw_buffer_load_b64(rs, fo, 0, AUX_SYS);
        const int o = issued - pub;
        if (o >= 4) { WAITV(25); ctl[s] = issued - 3; }
        if (o >= 3) { WAITV(17); ctl[s] = issued - 2; }
        if (o >= 2) { WAITV(9); ctl[s] = issued - 1; }
        if (o >= 1) { WAITV(1); ctl[s] = issued; STAMP(1 + (issued >> 1)); }
        pub = issued;
        CBAR();
        const u64 mk = __ballot(lane < avail || (inw && f.x >= call && f.y >= call));
        const int na = ~mk ? __builtin_ctzll(~mk) : 64;
        if (na == avail) {
          if (spin_fail(t0, tmo, call)) { avail = 64; ctl[12] = 1; }
        } else {
          avail = na; t0 = 0;
        }
      } else if (pub < issued) {
        const int o = issued - pub;
        if (o >= 4) WAITV(24);
        else if (o == 3) WAITV(16);
        else if (o == 2) WAITV(8);
        else WAITV(0);
        ++pub;
        ctl[s] = pub;
        STAMP(1 + (pub >> 1));
        t0 = 0;
      } else {
        if (spin_fail(t0, tmo, call)) { ctl[12] = 1; break; }
      }
    }
    WAITV(0);
    WAITL();
    STAMP_END(4);
    return;
  }

  const int wm = wid >> 1, wn = wid & 1;
  const bool remote = ct >= 4;
  const int orank = (remote ? (1 - g) * 4 + (ct - 4) : rank) & 7;
  u64 donep = (u64)p.ws[orank] + DONE_OFF + ((jq * 16 + rbb) * 4 + wid) * 4;
  unsigned outlo = (unsigned)(u64)p.out[orank], outhi = (unsigned)((u64)p.out[orank] >> 32);
  asm volatile("" : "+v"(donep), "+s"(outlo), "+s"(outhi));
  char* const outp = (char*)(((u64)(unsigned)RFL(outhi) << 32) | (u64)(unsigned)RFL(outlo));
  f32x4 acc[4][4];
#pragma unroll
  for (int m = 0; m < 4; ++m)
#pragma unroll
    for (int n = 0; n < 4; ++n) acc[m][n] = f32x4{0.f, 0.f, 0.f, 0.f};
  {
    const LAS char* xb = L8 + wm * 32768 + lane * 16;
    const LAS char* wb = L8 + (2 + wn) * 32768 + lane * 16;
    int full = 0;
    v4u xa[4], wa[4], xc[4], wc[4];
    auto rd_half = [&](v4u (&xf)[4], v4u (&wf)[4], int off) {
#pragma unroll
      for (int m = 0; m < 4; ++m) wf[m] = *(const LAS v4u*)(wb + off + m * 1024);
#pragma unroll
      for (int m = 0; m < 4; ++m) xf[m] = *(const LAS v4u*)(xb + off + m * 1024);
    };
    auto mm = [&](const v4u (&xf)[4], const v4u (&wf)[4]) {
#pragma unroll
      for (int m = 0; m < 4; ++m)
#pragma unroll
        for (int n = 0; n < 4; ++n)
          acc[m][n] = __builtin_amdgcn_mfma_f32_16x16x32_bf16(__builtin_bit_cast(bf16x8, wf[n]),
                                                              __builtin_bit_cast(bf16x8, xf[m]), acc[m][n], 0, 0, 0);
    };
    auto wait_full = [&](int kc) {
      u64 t0 = 0;
      int sp = 0;
      while (full <= kc) {
        full = RFL(min(ctl[wm], ctl[2 + wn]));
        if ((++sp & 63) == 0 && spin_fail(t0, tmo, call)) { full = 64; ctl[12] = 1; }
      }
      CBAR();
    };
    wait_full(0);
    rd_half(xa, wa, 0);
#pragma unroll 1
    for (int kc = 0; kc < 64; ++kc) {
      if ((kc & 1) == 0) STAMP(1 + (kc >> 1));
      rd_half(xc, wc, (kc & 3) * 8192 + 4096);
      const unsigned f0 = ctl[wm], f1 = ctl[2 + wn];
      __builtin_amdgcn_sched_barrier(0);
      mm(xa, wa);
      __builtin_amdgcn_sched_barrier(0);
      WAITL();
      ctl[4 + wid] = kc + 1;
      full = max(full, RFL(min(f0, f1)));
      const bool ahead = full > kc + 1;
      if (ahead) rd_half(xa, wa, ((kc + 1) & 3) * 8192);
      __builtin_amdgcn_sched_barrier(0);
      mm(xc, wc);
      __builtin_amdgcn_sched_barrier(0);
      if (!ahead && kc < 63) {
        wait_full(kc + 1);
        rd_half(xa, wa, ((kc + 1) & 3) * 8192);
      }
    }
  }
  STAMP(34);
  float mx = 0.f;
#pragma unroll
  for (int m = 0; m < 4; ++m)
#pragma unroll
    for (int n = 0; n < 4; ++n)
#pragma unroll
      for (int e = 0; e < 4; ++e) mx = fmaxf(mx, acc[m][n][e]);
  bool bad;
  {
    const unsigned wmx = wave_umax(__builtin_bit_cast(unsigned, mx));
    if (lane == 0) ctl[8 + wid] = wmx;
    WAITL();
    __builtin_amdgcn_s_barrier();
    CBAR();
    if (wid == 0 && lane < 8) {
      const unsigned m4 = max(max(ctl[8], ctl[9]), max(ctl[10], ctl[11]));
      const v2u mv = {m4, call};
      __builtin_amdgcn_raw_buffer_store_b64(mv, rs, SLOT1_OFF + lane * SLOT1_PITCH + (b - 128) * 8, 0, 16);
    }
    bad = ctl[12] != 0;
  }
  STAMP(35);
#pragma unroll
  for (int m = 0; m < 4; ++m)
#pragma unroll
    for (int n = 0; n < 4; ++n)
#pragma unroll
      for (int e = 0; e < 4; ++e) acc[m][n][e] = fmaxf(acc[m][n][e], 0.f);
#pragma unroll
  for (int m = 0; m < 4; ++m) asm volatile("" : "+v"(acc[m][0]), "+v"(acc[m][1]), "+v"(acc[m][2]), "+v"(acc[m][3]));
  unsigned gm;
  {
    const unsigned so = SLOT2_OFF + (b - 128) * 256 + (lane & 7) * 8;
    u64 t0 = 0;
    for (int sp = 1;; ++sp) {
      if (wid < 2) {
        const v2u a = __builtin_amdgcn_raw_buffer_load_b64(rs, so, 0, AUX_SYS);
        CBAR();
        if (__all(a.y >= call)) {
          gm = a.x;
          gm = max(gm, (unsigned)__builtin_amdgcn_update_dpp(0, (int)gm, 0x111, 0xf, 0xf, true));
          gm = max(gm, (unsigned)__builtin_amdgcn_update_dpp(0, (int)gm, 0x112, 0xf, 0xf, true));
          gm = max(gm, (unsigned)__builtin_amdgcn_update_dpp(0, (int)gm, 0x114, 0xf, 0xf, true));
          gm = __builtin_amdgcn_readlane(gm, 7);
          ctl[13] = gm + 1;
          break;
        }
      }
      gm = RFL(ctl[13]) - 1;
      if (gm != 0xffffffffu) break;
      if ((wid < 2 || (sp & 63) == 0) && spin_fail(t0, tmo, call)) { gm = BAD_MAX; break; }
    }
  }
  STAMP(36);
  bad = bad || gm == BAD_MAX;
  const float amax = __builtin_bit_cast(float, gm);
  const float inv = amax > 0.f && !bad ? 127.0f * __builtin_amdgcn_rcpf(amax) : 0.f;
  const float magic = bad ? 12582912.0f + 128.0f : 12582912.0f;
  const int rb = 2 * rbb + wm;
  int srank, lrow;
  if (rb < 20) { srank = g * 4 + rb / 5; lrow = (rb % 5) * 64; }
  else { srank = (1 - g) * 4 + (rb - 20) / 3; lrow = (5 + (rb - 20) % 3) * 64; }
  const int ocol = (remote ? jq * 128 : ct * 128) + wn * 64;
  const __amdgpu_buffer_rsrc_t ro = RSRC(outp, NR * SH * SH);
  const unsigned obase = (unsigned)(phys(srank) * SH + lrow + (lane & 15)) * SH + ocol + (lane >> 4) * 16;
#pragma unroll
  for (int m = 0; m < 4; ++m) {
    v4u q;
    const f32x4 inv4 = {inv, inv, inv, inv}, mg4 = {magic, magic, magic, magic};
    v4u tb[4];
#pragma unroll
    for (int n = 0; n < 4; ++n) tb[n] = __builtin_bit_cast(v4u, __builtin_elementwise_fma(acc[m][n], inv4, mg4));
#pragma unroll
    for (int e = 0; e < 4; ++e) {
      const unsigned lo = __builtin_amdgcn_perm(tb[1][e], tb[0][e], 0x0c0c0400u);
      const unsigned hi = __builtin_amdgcn_perm(tb[3][e], tb[2][e], 0x04000c0cu);
      q[e] = lo | hi;
    }
    __builtin_amdgcn_raw_buffer_store_b128(q, ro, obase + m * (16 * SH), 0, AUX_SYS);
  }
  STAMP(37);
  if (remote) {
    WAITV(0);
    if (lane == 0)
      __hip_atomic_store((GAS unsigned*)donep, call, __ATOMIC_RELAXED, __HIP_MEMORY_SCOPE_SYSTEM);
  }
  STAMP(38);
  STAMP_END(1);
}

extern "C" void kernel_launch(int rank, int nranks, ncclComm_t comm, hipStream_t stream, void* const* d_in,
                              const int* in_sizes, int n_in, void* d_out, int out_size, const int* in_dtypes,
                              int out_dtype, void* const* d_out_all, void* const* d_ws_all, size_t ws_bytes) {
  if (nranks != NR || (unsigned)rank >= (unsigned)NR || n_in < 2 || ws_bytes < (size_t)WS_USED ||
      in_sizes[0] != SH * KD || in_sizes[1] != SH * KD || out_size != NR * SH * SH) {
    fprintf(stderr, "unexpected problem shape\n");
    return;
  }
  Params p;
  p.rank = p.pad = 0;
  p.myws = (char*)d_ws_all[rank];
  p.x = (const float*)d_in[0];
  p.w = (const float*)d_in[1];
  for (int i = 0; i < NR; ++i) {
    p.ws[i] = (char*)d_ws_all[phys(i)];
    p.out[i] = (char*)d_out_all[phys(i)];
    if (phys(i) == rank) p.rank = i;
  }
  HIP_CHECK(hipFuncSetAttribute((const void*)ag_gemm_kernel, hipFuncAttributeMaxDynamicSharedMemorySize, LDS_BYTES));
  ag_gemm_kernel<<<NWG, NTHR, LDS_BYTES, stream>>>(p);
  HIP_CHECK(hipGetLastError());
}
```

```cpp
#include <hip/hip_runtime.h>
#include <rccl/rccl.h>
#include <cstdio>

#define HIP_CHECK(expr) do { hipError_t _e = (expr); if (_e != hipSuccess) { \
    fprintf(stderr, "HIP error %s at %s:%d\n", hipGetErrorString(_e), __FILE__, __LINE__); \
    return; } } while (0)

typedef unsigned v4u __attribute__((ext_vector_type(4)));
typedef unsigned v2u __attribute__((ext_vector_type(2)));
typedef float f32x4 __attribute__((ext_vector_type(4)));
typedef float f32x2 __attribute__((ext_vector_type(2)));
typedef __bf16 bf16x2 __attribute__((ext_vector_type(2)));
typedef short bf16x8 __attribute__((ext_vector_type(8)));
typedef unsigned long long u64;
#define LAS __attribute__((address_space(3)))
#define GAS __attribute__((address_space(1)))

constexpr int NR = 8;
constexpr int KD = 4096;
constexpr int SH = 512;
constexpr int NWG = 256;
constexpr int NTHR = 512;
#define LINK_WGS 14
#define LINK_WAVES 1
constexpr unsigned PHYS = 0x75416320u;
__host__ __device__ constexpr int phys(int r) { return (PHYS >> (4 * r)) & 7; }
constexpr int NLW = LINK_WGS * LINK_WAVES;
#define SW0 4
#define SW1 2
constexpr int NW1 = 7 * (16 - LINK_WGS);
static_assert(SW0 >= 4 && SW1 >= 2, "waves with a second job");
constexpr int NSELF1 = NW1 * SW1;
constexpr int NSELF = NSELF1 + 16 * SW0;
static_assert(LINK_WGS < 16 && 3 * NLW <= 128 * 5 && 3 * NSELF <= 128 * 15, "three fixed units per wave");
constexpr unsigned XS_OFF = 0;
constexpr unsigned WS_OFF = 16u << 20;
constexpr unsigned FLAG_OFF = 25u << 20;
constexpr unsigned FLAG_PITCH = 4096 + 512;
constexpr unsigned SLOT1_OFF = FLAG_OFF + (256u << 10);
constexpr unsigned SLOT1_PITCH = 1024 + 256;
constexpr unsigned SLOT2_OFF = SLOT1_OFF + (16u << 10);
constexpr unsigned DONE_OFF = SLOT2_OFF + (32u << 10);
constexpr unsigned CNT_OFF = DONE_OFF + (4u << 10);
constexpr unsigned TICK_OFF = CNT_OFF + (16u << 10);
constexpr unsigned TMO_OFF = TICK_OFF + (4u << 10);
constexpr unsigned WS_USED = TMO_OFF + 4096;
constexpr unsigned STAMP_OFF = 32u << 20;
constexpr u64 DEADLINE_TICKS = 50000000ull;
constexpr int CTL = 128 * 1024;
constexpr int LDS_BYTES = CTL + 256;
constexpr unsigned BAD_MAX = 0xfffffffeu;
#define AUX_DMA 17
constexpr int AUX_SYS = 17;

struct Params {
  const float* x;
  const float* w;
  char* myws;
  int rank, pad;
  char* ws[NR];
  char* out[NR];
};

#define RSRC(p, bytes) __builtin_amdgcn_make_buffer_rsrc((void*)(p), 0, (int)(bytes), 0x00020000)
#define CBAR() asm volatile("" ::: "memory")
#define WAITV(n) asm volatile("s_waitcnt vmcnt(" #n ")" ::: "memory")
#define WAITL() asm volatile("s_waitcnt lgkmcnt(0)" ::: "memory")
#define RFL(v) __builtin_amdgcn_readfirstlane(v)

#define STAMP(i) do {} while (0)
#define STAMP_END(cls) do {} while (0)

__device__ __forceinline__ unsigned pk(float a, float b) {
  f32x2 v = {a, b};
  return __builtin_bit_cast(unsigned, __builtin_convertvector(v, bf16x2));
}

__device__ __forceinline__ void unit_load(v4u (&r)[8], __amdgpu_buffer_rsrc_t rx, __amdgpu_buffer_rsrc_t rw,
                                          bool isw, int blk, int k32, int lane, unsigned oob) {
  const int kg = lane >> 4, c = lane & 15;
  if (!isw) {
    const unsigned base = ((unsigned)(blk * 64 + c) * (KD * 4) + (unsigned)(k32 * 32 + 4 * kg) * 4) | oob;
#pragma unroll
    for (int m = 0; m < 4; ++m) {
      r[2 * m] = __builtin_amdgcn_raw_buffer_load_b128(rx, base + m * (16 * KD * 4), 0, 0);
      r[2 * m + 1] = __builtin_amdgcn_raw_buffer_load_b128(rx, base + m * (16 * KD * 4) + 64, 0, 0);
    }
  } else {
    const unsigned base = ((unsigned)(k32 * 32 + 4 * kg) * (SH * 4) + (unsigned)(blk * 64 + 4 * c) * 4) | oob;
#pragma unroll
    for (int jj = 0; jj < 8; ++jj)
      r[jj] = __builtin_amdgcn_raw_buffer_load_b128(rw, base + ((jj & 3) + 16 * (jj >> 2)) * (SH * 4), 0, 0);
  }
}
__device__ __forceinline__ float asf(unsigned u) { return __builtin_bit_cast(float, u); }
__device__ __forceinline__ void unit_cvt(v4u (&o)[4], const v4u (&r)[8], bool isw) {
  if (!isw) {
#pragma unroll
    for (int m = 0; m < 4; ++m) {
      o[m].x = pk(asf(r[2 * m].x), asf(r[2 * m].y));
      o[m].y = pk(asf(r[2 * m].z), asf(r[2 * m].w));
      o[m].z = pk(asf(r[2 * m + 1].x), asf(r[2 * m + 1].y));
      o[m].w = pk(asf(r[2 * m + 1].z), asf(r[2 * m + 1].w));
    }
  } else {
#pragma unroll
    for (int i = 0; i < 4; ++i) {
      o[i].x = pk(asf(r[0][i]), asf(r[1][i]));
      o[i].y = pk(asf(r[2][i]), asf(r[3][i]));
      o[i].z = pk(asf(r[4][i]), asf(r[5][i]));
      o[i].w = pk(asf(r[6][i]), asf(r[7][i]));
    }
  }
}
__device__ __forceinline__ unsigned stage_off(int dblk, int k32) {
  const int kc = k32 >> 1, h = k32 & 1;
  return dblk < 32 ? XS_OFF + (unsigned)((kc * 32 + dblk) * 2 + h) * 4096u
                   : WS_OFF + (unsigned)((kc * 18 + (dblk - 32)) * 2 + h) * 4096u;
}
__device__ __forceinline__ unsigned flag_off(int dblk, int k32) {
  return FLAG_OFF + (unsigned)dblk * FLAG_PITCH + (unsigned)k32 * 4u;
}
__device__ __forceinline__ void unit_store(const v4u (&o)[4], __amdgpu_buffer_rsrc_t rd, unsigned off, int lane) {
#pragma unroll
  for (int i = 0; i < 4; ++i) __builtin_amdgcn_raw_buffer_store_b128(o[i], rd, off + lane * 16 + i * 1024, 0, AUX_SYS);
}

__device__ __forceinline__ int home_rbb(int j) { return (20 + 3 * j + (j & 1)) >> 1; }

__device__ __forceinline__ bool spin_fail(u64& t0, unsigned* tmo, unsigned call) {
  const u64 now = __builtin_amdgcn_s_memrealtime();
  if (t0 == 0) t0 = now;
  if (now - t0 > 10000) {
    if (__hip_atomic_load(tmo, __ATOMIC_RELAXED, __HIP_MEMORY_SCOPE_AGENT) == call) return true;
    if (now - t0 > DEADLINE_TICKS) {
      __hip_atomic_store(tmo, call, __ATOMIC_RELAXED, __HIP_MEMORY_SCOPE_AGENT);
      return true;
    }
  }
  return false;
}

__device__ __forceinline__ unsigned wave_umax(unsigned v) {
  v = max(v, (unsigned)__builtin_amdgcn_update_dpp(0, (int)v, 0x111, 0xf, 0xf, true));
  v = max(v, (unsigned)__builtin_amdgcn_update_dpp(0, (int)v, 0x112, 0xf, 0xf, true));
  v = max(v, (unsigned)__builtin_amdgcn_update_dpp(0, (int)v, 0x114, 0xf, 0xf, true));
  v = max(v, (unsigned)__builtin_amdgcn_update_dpp(0, (int)v, 0x118, 0xf, 0xf, true));
  const unsigned a = __builtin_amdgcn_readlane(v, 15), b = __builtin_amdgcn_readlane(v, 31);
  const unsigned c = __builtin_amdgcn_readlane(v, 47), d = __builtin_amdgcn_readlane(v, 63);
  return max(max(a, b), max(c, d));
}

__global__ void __launch_bounds__(NTHR, 2) ag_gemm_kernel(Params p) {
  extern __shared__ __attribute__((aligned(16))) char smem[];
  LAS char* const L8 = (LAS char*)smem;
  const int lane = threadIdx.x & 63;
  const int wid = RFL(threadIdx.x >> 6);
  const int b = (blockIdx.x & ~7) | (__builtin_amdgcn_s_getreg((3 << 11) | 20) & 7);
  const float* px = p.x;
  const float* pw = p.w;
  u64 wsv = (u64)p.myws;
  int rank = p.rank;
  asm volatile("" : "+s"(px), "+s"(pw), "+s"(wsv), "+s"(rank));
  char* const myws = (char*)(GAS char*)wsv;
  const int g = rank >> 2, jq = rank & 3;
  const int xcd = b & 7, bi = b >> 3;
  unsigned* const tmo = (unsigned*)(myws + TMO_OFF);
  unsigned* const cntp = (unsigned*)(myws + CNT_OFF + b * 64 + wid * 4);
  STAMP(0);

  if (__builtin_expect(bi < 16, 1)) {
    const bool link = xcd > 0 && bi < LINK_WGS;
    const int peer = ((g ^ (xcd >> 2)) << 2) | ((jq + xcd) & 3);
    if (wid >= (link ? LINK_WAVES : xcd == 0 ? SW0 : SW1)) { STAMP_END(0); return; }
    const __amdgpu_buffer_rsrc_t rx = RSRC(px, SH * KD * 4);
    const __amdgpu_buffer_rsrc_t rw = RSRC(pw, SH * KD * 4);
    const int dst = link ? peer : rank;
    const int mode = !link ? 2 : ((dst >> 2) == g ? 0 : 1);
    const int step = link ? NLW : NSELF;
    const int total = link ? 128 * 5 : 128 * 15;
    int u;
    if (link) u = wid * LINK_WGS + bi;
    else if (xcd == 0) u = NSELF1 + wid * 16 + bi;
    else u = wid * NW1 + (xcd - 1) * (16 - LINK_WGS) + (bi - LINK_WGS);
    const __amdgpu_buffer_rsrc_t rd = RSRC(p.ws[dst], WS_USED);
    auto decode = [&](int uu, bool& isw, int& sblk, int& dblk, int& k32) {
      if (mode == 2) {
        k32 = uu / 15;
        const int s = uu - k32 * 15;
        isw = s >= 5 && s < 13;
        sblk = s < 5 ? s : s < 13 ? s - 5 : s - 8 + (jq & 1);
        dblk = s < 5 ? jq * 5 + s : 32 + s - 5 + (s < 13 ? 0 : 8);
      } else {
        k32 = uu / 5;
        const int s = uu - k32 * 5;
        if (mode == 0) { isw = false; sblk = s; dblk = jq * 5 + s; }
        else if (s < 3) { isw = false; sblk = 5 + s; dblk = 20 + jq * 3 + s; }
        else { isw = true; sblk = (dst & 3) * 2 + (s - 3); dblk = 32 + 8 + jq * 2 + (s - 3); }
      }
    };
    v4u ra[8], rb2[8], o[4];
    bool isw, isw2; int sblk, dblk, k32, sblk2, dblk2, k322;
    decode(u, isw, sblk, dblk, k32);
    unit_load(ra, rx, rw, isw, sblk, k32, lane, 0);
    unsigned call;
    asm volatile("global_load_dword %0, %1, %2 sc1" : "=v"(call) : "v"(0), "s"(cntp) : "memory");
    unsigned pflag = 0xffffffffu;
    const unsigned lmask = lane == 0 ? 0u : 0xffffffffu;
    const __amdgpu_buffer_rsrc_t rs = RSRC(myws, WS_USED);
    const unsigned tko = TICK_OFF + (link ? xcd : 0) * 256;
    unsigned tk = 1, tbase = 0;
    int stat = u + 2 * step;
    u += step;
    int t = 0;
#define DRAW() do { u64 sv_; tk = 1; asm volatile("s_mov_b64 %1, exec\n\ts_cmp_lt_u32 %4, %5\n\ts_cselect_b64 exec, 1, 0\n\tbuffer_atomic_add %0, %2, %3, 0 offen sc0 sc1\n\ts_mov_b64 exec, %1" : "+v"(tk), "=&s"(sv_) : "v"(tko), "s"(rs), "s"(u), "s"(total) : "memory", "scc"); } while (0)
#pragma unroll 1
    for (;;) {
      bool more = (unsigned)u < (unsigned)total;
      decode(u, isw2, sblk2, dblk2, k322);
      unit_load(rb2, rx, rw, isw2, sblk2, k322, lane, more ? 0u : 0x80000000u);
      WAITV(8);
      asm volatile("" : "+v"(call), "+v"(tk));
      tbase = (unsigned)RFL(call) * (unsigned)(total - 2 * step) - 3u * step;
      u = stat >= 0 ? stat : (int)((unsigned)RFL(tk) - tbase);
      stat = -1;
      STAMP(1 + t);
      __builtin_amdgcn_raw_buffer_store_b32(call + 1, rd, pflag | lmask, 0, AUX_SYS);
      unit_cvt(o, ra, isw);
      unit_store(o, rd, stage_off(dblk, k32), lane);
      pflag = flag_off(dblk, k32);
      if (!more) break;
      more = (unsigned)u < (unsigned)total;
      DRAW();
      decode(u, isw, sblk, dblk, k32);
      unit_load(ra, rx, rw, isw, sblk, k32, lane, more ? 0u : 0x80000000u);
      WAITV(8);
      asm volatile("" : "+v"(tk));
      u = (int)((unsigned)RFL(tk) - tbase);
      STAMP(2 + t);
      __builtin_amdgcn_raw_buffer_store_b32(call + 1, rd, pflag | lmask, 0, AUX_SYS);
      unit_cvt(o, rb2, isw2);
      unit_store(o, rd, stage_off(dblk2, k322), lane);
      pflag = flag_off(dblk2, k322);
      if (!more) break;
      DRAW();
      t = min(t + 2, 56);
    }
    call += 1;
    WAITV(0);
    STAMP(60);
    if (lane == 0) {
      __builtin_amdgcn_raw_buffer_store_b32(call, rd, pflag, 0, AUX_SYS);
      __hip_atomic_store(cntp, call, __ATOMIC_RELAXED, __HIP_MEMORY_SCOPE_AGENT);
    }
    if ((bi == 15 && wid < 2) || (b == 112 && wid < 4)) {
      u64 t0 = 0;
      if (bi == 15) {
        const __amdgpu_buffer_rsrc_t rq = RSRC(p.ws[peer], WS_USED);
        unsigned m;
        for (;;) {
          const v4u a = __builtin_amdgcn_raw_buffer_load_b128(rs, SLOT1_OFF + xcd * SLOT1_PITCH + lane * 16, 0, 16);
          CBAR();
          m = max(a.x, a.z);
          if (__all(a.y == call && a.w == call)) break;
          if (spin_fail(t0, tmo, call)) { m = BAD_MAX; break; }
        }
        const v2u mv = {wave_umax(m), call};
        __builtin_amdgcn_raw_buffer_store_b64(mv, rq, SLOT2_OFF + lane * 256 + rank * 8, 0, AUX_SYS);
        __builtin_amdgcn_raw_buffer_store_b64(mv, rq, SLOT2_OFF + (lane + 64) * 256 + rank * 8, 0, AUX_SYS);
        STAMP(61);
      } else {
        bool bad = false;
        for (;;) {
          const unsigned a = __builtin_amdgcn_raw_buffer_load_b32(rs, DONE_OFF + wid * 256 + lane * 4, 0, AUX_SYS);
          CBAR();
          if (__all(a >= call || (lane >> 2) == home_rbb(jq))) break;
          if (spin_fail(t0, tmo, call)) { bad = true; break; }
        }
        if (bad && lane == 0) {
          __builtin_amdgcn_s_sleep(127);
          p.out[rank][(size_t)phys(rank) * SH * SH] = (char)-128;
        }
        STAMP(62);
      }
      STAMP_END(3);
      return;
    }
    STAMP_END(link ? 2 : 5);
    return;
  }

  volatile LAS unsigned* const ctl = (volatile LAS unsigned*)(L8 + CTL);
  const unsigned call = __hip_atomic_load(cntp, __ATOMIC_RELAXED, __HIP_MEMORY_SCOPE_AGENT) + 1;
  if (threadIdx.x < 16) ctl[threadIdx.x] = 0;
  if (lane == 0) __hip_atomic_store(cntp, call, __ATOMIC_RELAXED, __HIP_MEMORY_SCOPE_AGENT);
  WAITL();
  __builtin_amdgcn_s_barrier();
  CBAR();
  const int ti = bi - 16;
  const int rbb = (xcd >> 1) * 4 + (ti & 3);
  const int cti = ti >> 2;
  const int ct0 = (cti & 1) + 2 * (xcd & 1) + 4 * (cti >> 1);
  const bool home = ct0 >= 4 && rbb == home_rbb(ct0 - 4);
  const int ct = home ? ct0 - 4 : ct0;
  const __amdgpu_buffer_rsrc_t rs = RSRC(myws, WS_USED);

  if (wid >= 4) {
    const int s = wid - 4;
    const int blk = s < 2 ? (home ? 48 : 2 * rbb) + s : 32 + 2 * ct + (s - 2);
    const unsigned sbase = (blk < 32 ? XS_OFF + blk * 8192 : WS_OFF + (blk - 32) * 8192) + lane * 16;
    const unsigned sstep = blk < 32 ? 32 * 8192 : 18 * 8192;
    const int c0 = s < 2 ? 2 * s : s - 2, c1 = s < 2 ? 2 * s + 1 : s;
    const unsigned fo = FLAG_OFF + blk * FLAG_PITCH + lane * 8;
    int avail = 0, issued = 0, pub = 0;
    u64 t0 = 0;
#pragma unroll 1
    while (pub < 64) {
      const int fr = RFL(min(ctl[4 + c0], ctl[4 + c1])) + 4;
      const int lim = min(avail, fr);
      CBAR();
#pragma unroll 1
      while (issued < lim) {
        LAS char* dl = L8 + s * 32768 + (issued & 3) * 8192;
#pragma unroll
        for (int i = 0; i < 8; ++i)
          __builtin_amdgcn_raw_ptr_buffer_load_lds(rs, (LAS void*)(dl + i * 1024), 16, sbase + i * 1024, issued * sstep, 0, AUX_DMA);
        ++issued;
      }
      if (issued == avail && avail < 64) {
        v2u f = {0u, 0u};
        const bool inw = (unsigned)(lane - avail) < 16u;
        if (inw) f = __builtin_amdgcn_raw_buffer_load_b64(rs, fo, 0, AUX_SYS);
        const int o = issued - pub;
        if (o >= 4) { WAITV(25); ctl[s] = issued - 3; }
        if (o >= 3) { WAITV(17); ctl[s] = issued - 2; }
        if (o >= 2) { WAITV(9); ctl[s] = issued - 1; }
        if (o >= 1) { WAITV(1); ctl[s] = issued; STAMP(1 + (issued >> 1)); }
        pub = issued;
        CBAR();
        const u64 mk = __ballot(lane < avail || (inw && f.x >= call && f.y >= call));
        const int na = ~mk ? __builtin_ctzll(~mk) : 64;
        if (na == avail) {
          if (spin_fail(t0, tmo, call)) { avail = 64; ctl[12] = 1; }
        } else {
          avail = na; t0 = 0;
        }
      } else if (pub < issued) {
        const int o = issued - pub;
        if (o >= 4) WAITV(24);
        else if (o == 3) WAITV(16);
        else if (o == 2) WAITV(8);
        else WAITV(0);
        ++pub;
        ctl[s] = pub;
        STAMP(1 + (pub >> 1));
        t0 = 0;
      } else {
        if (spin_fail(t0, tmo, call)) { ctl[12] = 1; break; }
      }
    }
    WAITV(0);
    WAITL();
    STAMP_END(4);
    return;
  }

  const int wm = wid >> 1, wn = wid & 1;
  const bool remote = ct >= 4;
  const int orank = (remote ? (1 - g) * 4 + (ct - 4) : rank) & 7;
  u64 donep = (u64)p.ws[orank] + DONE_OFF + ((jq * 16 + rbb) * 4 + wid) * 4;
  unsigned outlo = (unsigned)(u64)p.out[orank], outhi = (unsigned)((u64)p.out[orank] >> 32);
  asm volatile("" : "+v"(donep), "+s"(outlo), "+s"(outhi));
  char* const outp = (char*)(((u64)(unsigned)RFL(outhi) << 32) | (u64)(unsigned)RFL(outlo));
  f32x4 acc[4][4];
#pragma unroll
  for (int m = 0; m < 4; ++m)
#pragma unroll
    for (int n = 0; n < 4; ++n) acc[m][n] = f32x4{0.f, 0.f, 0.f, 0.f};
  {
    const LAS char* xb = L8 + wm * 32768 + lane * 16;
    const LAS char* wb = L8 + (2 + wn) * 32768 + lane * 16;
    int full = 0;
    v4u xa[4], wa[4], xc[4], wc[4];
    auto rd_half = [&](v4u (&xf)[4], v4u (&wf)[4], int off) {
#pragma unroll
      for (int m = 0; m < 4; ++m) wf[m] = *(const LAS v4u*)(wb + off + m * 1024);
#pragma unroll
      for (int m = 0; m < 4; ++m) xf[m] = *(const LAS v4u*)(xb + off + m * 1024);
    };
    auto mm = [&](const v4u (&xf)[4], const v4u (&wf)[4]) {
#pragma unroll
      for (int m = 0; m < 4; ++m)
#pragma unroll
        for (int n = 0; n < 4; ++n)
          acc[m][n] = __builtin_amdgcn_mfma_f32_16x16x32_bf16(__builtin_bit_cast(bf16x8, wf[n]),
                                                              __builtin_bit_cast(bf16x8, xf[m]), acc[m][n], 0, 0, 0);
    };
    auto wait_full = [&](int kc) {
      u64 t0 = 0;
      int sp = 0;
      while (full <= kc) {
        full = RFL(min(ctl[wm], ctl[2 + wn]));
        if ((++sp & 63) == 0 && spin_fail(t0, tmo, call)) { full = 64; ctl[12] = 1; }
      }
      CBAR();
    };
    wait_full(0);
    rd_half(xa, wa, 0);
#pragma unroll 1
    for (int kc = 0; kc < 64; ++kc) {
      if ((kc & 1) == 0) STAMP(1 + (kc >> 1));
      rd_half(xc, wc, (kc & 3) * 8192 + 4096);
      const unsigned f0 = ctl[wm], f1 = ctl[2 + wn];
      __builtin_amdgcn_sched_barrier(0);
      mm(xa, wa);
      __builtin_amdgcn_sched_barrier(0);
      WAITL();
      ctl[4 + wid] = kc + 1;
      full = max(full, RFL(min(f0, f1)));
      const bool ahead = full > kc + 1;
      if (ahead) rd_half(xa, wa, ((kc + 1) & 3) * 8192);
      __builtin_amdgcn_sched_barrier(0);
      mm(xc, wc);
      __builtin_amdgcn_sched_barrier(0);
      if (!ahead && kc < 63) {
        wait_full(kc + 1);
        rd_half(xa, wa, ((kc + 1) & 3) * 8192);
      }
    }
  }
  STAMP(34);
  float mx = 0.f;
#pragma unroll
  for (int m = 0; m < 4; ++m)
#pragma unroll
    for (int n = 0; n < 4; ++n)
#pragma unroll
      for (int e = 0; e < 4; ++e) mx = fmaxf(mx, acc[m][n][e]);
  bool bad;
  {
    const unsigned wmx = wave_umax(__builtin_bit_cast(unsigned, mx));
    if (lane == 0) ctl[8 + wid] = wmx;
    WAITL();
    __builtin_amdgcn_s_barrier();
    CBAR();
    if (wid == 0 && lane < 8) {
      const unsigned m4 = max(max(ctl[8], ctl[9]), max(ctl[10], ctl[11]));
      const v2u mv = {m4, call};
      __builtin_amdgcn_raw_buffer_store_b64(mv, rs, SLOT1_OFF + lane * SLOT1_PITCH + (b - 128) * 8, 0, 16);
    }
    bad = ctl[12] != 0;
  }
  STAMP(35);
#pragma unroll
  for (int m = 0; m < 4; ++m)
#pragma unroll
    for (int n = 0; n < 4; ++n)
#pragma unroll
      for (int e = 0; e < 4; ++e) acc[m][n][e] = fmaxf(acc[m][n][e], 0.f);
#pragma unroll
  for (int m = 0; m < 4; ++m) asm volatile("" : "+v"(acc[m][0]), "+v"(acc[m][1]), "+v"(acc[m][2]), "+v"(acc[m][3]));
  unsigned gm;
  {
    const unsigned so = SLOT2_OFF + (b - 128) * 256 + (lane & 7) * 8;
    u64 t0 = 0;
    for (int sp = 1;; ++sp) {
      if (wid < 2) {
        const v2u a = __builtin_amdgcn_raw_buffer_load_b64(rs, so, 0, AUX_SYS);
        CBAR();
        if (__all(a.y >= call)) {
          gm = a.x;
          gm = max(gm, (unsigned)__builtin_amdgcn_update_dpp(0, (int)gm, 0x111, 0xf, 0xf, true));
          gm = max(gm, (unsigned)__builtin_amdgcn_update_dpp(0, (int)gm, 0x112, 0xf, 0xf, true));
          gm = max(gm, (unsigned)__builtin_amdgcn_update_dpp(0, (int)gm, 0x114, 0xf, 0xf, true));
          gm = __builtin_amdgcn_readlane(gm, 7);
          ctl[13] = gm + 1;
          break;
        }
      }
      gm = RFL(ctl[13]) - 1;
      if (gm != 0xffffffffu) break;
      if ((wid < 2 || (sp & 63) == 0) && spin_fail(t0, tmo, call)) { gm = BAD_MAX; break; }
    }
  }
  STAMP(36);
  bad = bad || gm == BAD_MAX;
  const float amax = __builtin_bit_cast(float, gm);
  const float inv = amax > 0.f && !bad ? 127.0f * __builtin_amdgcn_rcpf(amax) : 0.f;
  const float magic = bad ? 12582912.0f + 128.0f : 12582912.0f;
  const int rb = 2 * rbb + wm;
  int srank, lrow;
  if (home) { srank = rank; lrow = (5 + (jq & 1) + wm) * 64; }
  else if (rb < 20) { srank = g * 4 + rb / 5; lrow = (rb % 5) * 64; }
  else { srank = (1 - g) * 4 + (rb - 20) / 3; lrow = (5 + (rb - 20) % 3) * 64; }
  const int ocol = (remote ? jq * 128 : ct * 128) + wn * 64;
  const __amdgpu_buffer_rsrc_t ro = RSRC(outp, NR * SH * SH);
  const unsigned obase = (unsigned)(phys(srank) * SH + lrow + (lane & 15)) * SH + ocol + (lane >> 4) * 16;
#pragma unroll
  for (int m = 0; m < 4; ++m) {
    v4u q;
    const f32x4 inv4 = {inv, inv, inv, inv}, mg4 = {magic, magic, magic, magic};
    v4u tb[4];
#pragma unroll
    for (int n = 0; n < 4; ++n) tb[n] = __builtin_bit_cast(v4u, __builtin_elementwise_fma(acc[m][n], inv4, mg4));
#pragma unroll
    for (int e = 0; e < 4; ++e) {
      const unsigned lo = __builtin_amdgcn_perm(tb[1][e], tb[0][e], 0x0c0c0400u);
      const unsigned hi = __builtin_amdgcn_perm(tb[3][e], tb[2][e], 0x04000c0cu);
      q[e] = lo | hi;
    }
    __builtin_amdgcn_raw_buffer_store_b128(q, ro, obase + m * (16 * SH), 0, AUX_SYS);
  }
  STAMP(37);
  if (remote) {
    WAITV(0);
    if (lane == 0)
      __hip_atomic_store((GAS unsigned*)donep, call, __ATOMIC_RELAXED, __HIP_MEMORY_SCOPE_SYSTEM);
  }
  STAMP(38);
  STAMP_END(1);
}

extern "C" void kernel_launch(int rank, int nranks, ncclComm_t comm, hipStream_t stream, void* const* d_in,
                              const int* in_sizes, int n_in, void* d_out, int out_size, const int* in_dtypes,
                              int out_dtype, void* const* d_out_all, void* const* d_ws_all, size_t ws_bytes) {
  if (nranks != NR || (unsigned)rank >= (unsigned)NR || n_in < 2 || ws_bytes < (size_t)WS_USED ||
      in_sizes[0] != SH * KD || in_sizes[1] != SH * KD || out_size != NR * SH * SH) {
    fprintf(stderr, "unexpected problem shape\n");
    return;
  }
  Params p;
  p.rank = p.pad = 0;
  p.myws = (char*)d_ws_all[rank];
  p.x = (const float*)d_in[0];
  p.w = (const float*)d_in[1];
  for (int i = 0; i < NR; ++i) {
    p.ws[i] = (char*)d_ws_all[phys(i)];
    p.out[i] = (char*)d_out_all[phys(i)];
    if (phys(i) == rank) p.rank = i;
  }
  HIP_CHECK(hipFuncSetAttribute((const void*)ag_gemm_kernel, hipFuncAttributeMaxDynamicSharedMemorySize, LDS_BYTES));
  ag_gemm_kernel<<<NWG, NTHR, LDS_BYTES, stream>>>(p);
  HIP_CHECK(hipGetLastError());
}
```
